```python
import numpy as np

kernel_name = "gpu_ag_gemm_m4096_k4096_n32768_bf16_silu"


ABSMAX_RTOL = 0.008

COMPUTE_DTYPE = "bfloat16"
EPILOGUE = "silu"
IN_QUANT = None
OUT_QUANT = None
IN_DTYPES = {"x": "bfloat16", "w_mat": "bfloat16"}
WORLD_SIZE = 8

def _to_bf16(x):
    u = np.ascontiguousarray(x, dtype=np.float32).view(np.uint32)
    return ((u + 0x7FFF + ((u >> 16) & 1)) & 0xFFFF0000).view(np.float32)

def setup_inputs():

    x = _to_bf16(np.random.randn(4096, 4096).astype(np.float32))
    w_mat = _to_bf16((np.random.randn(4096, 32768) / np.sqrt(4096)).astype(np.float32))
    return {"x": x, "w_mat": w_mat}

def _epilogue(y):
    return (y / (1.0 + np.exp(-np.clip(y, -60.0, 60.0)))).astype(np.float32)

def reference(x, w_mat):
    y = (x @ w_mat).astype(np.float32)
    return _epilogue(y)
```

```cpp
#include <hip/hip_runtime.h>
#include <rccl/rccl.h>
#include <cstdio>
#define LAS __attribute__((address_space(3)))
typedef unsigned short bf16_t;
typedef short bf16x8 __attribute__((ext_vector_type(8)));
typedef short s16x4 __attribute__((ext_vector_type(4)));
typedef float f32x4 __attribute__((ext_vector_type(4)));
typedef unsigned u32x4 __attribute__((ext_vector_type(4)));
typedef unsigned u32x2 __attribute__((ext_vector_type(2)));

constexpr int MM = 4096, NN = 4096, KK = 4096;
constexpr int BM = 256, BK = 64, HALF = 128, HTB = HALF * BK * 2, STAGE_BYTES = 8 * HTB, NXCD = 8, WGM = 8;

__host__ __device__ __forceinline__ int lds_byte(int r, int c) { const int st = (r >> 4) * 2 + (c >> 5), rr = r & 15, cc = c & 31, ob = rr * 64 + cc * 2; return st * 1024 + (ob ^ (((ob >> 9) & 1) << 5)); }
__host__ __device__ __forceinline__ void stage_rc(int b, int& R, int& C) { const int st = b / 1024, sb = b % 1024, swz = sb ^ (((sb >> 9) & 1) << 5); R = (st >> 1) * 16 + swz / 64; C = (st & 1) * 32 + (swz % 64) / 2; }
__host__ __device__ __forceinline__ int bswz(int row) { return ((row & 3) | (((row >> 3) & 1) << 2)) << 1; }

__device__ __forceinline__ unsigned cvt_pk_bf16(float lo, float hi) { unsigned r; asm volatile("v_cvt_pk_bf16_f32 %0, %1, %2" : "=v"(r) : "v"(lo), "v"(hi)); return r; }
__device__ __forceinline__ float silu_f(float y) {
    const float c = fminf(fmaxf(y, -60.f), 60.f);
    const float e = __builtin_amdgcn_exp2f(c * -1.44269504088896340736f);
    return y * __builtin_amdgcn_rcpf(1.0f + e);
}

__global__ __launch_bounds__(512, 2) void gemm_k(const bf16_t* __restrict__ A, const bf16_t* __restrict__ W, bf16_t* __restrict__ O) {
    extern __shared__ __attribute__((aligned(16))) unsigned char shm[];
    LAS unsigned char* lds = (LAS unsigned char*)shm;
    const int tid = threadIdx.x, wid = __builtin_amdgcn_readfirstlane(tid >> 6), lane = tid & 63, wr = wid >> 2, wc = wid & 3, fr = lane & 15, fq = lane >> 4;
    constexpr int K = KK, N = NN, nt = K / BK, nM = MM / BM, nN = NN / BM, nwg = nM * nN;
    int wgid = blockIdx.x; { const int q = nwg / NXCD, xcd = wgid % NXCD, off = wgid / NXCD; wgid = xcd * q + off; }
    const int nig = WGM * nN, gid = wgid / nig, fm = gid * WGM;
    const int pm = fm + ((wgid % nig) % WGM), pn = (wgid % nig) / WGM;

#define SA(b, h) (((b) * 2 + (h)) * HTB)
#define SB(b, h) ((4 + (b) * 2 + (h)) * HTB)
    unsigned voffA[2], voffB;
#pragma unroll
    for (int i = 0; i < 2; ++i) { int R, C; stage_rc(tid * 16 + i * 8192, R, C); voffA[i] = (unsigned)(R * K + C) * 2u; }
    { const int b = tid * 16, row = b >> 8, chp = (b >> 4) & 15, ch = chp ^ bswz(row); voffB = (unsigned)(row * N * 2 + ch * 16); }
    const size_t kstepA = (size_t)(BK * 2), kstepB = (size_t)BK * N * 2;
    const size_t hstepA = (size_t)HALF * K * 2, hstepB = (size_t)HALF * 2;
    const unsigned ldsw = (unsigned)wid * 1024u;
    const int aoff = lds_byte(wr * 64 + fr, fq * 8);
    int boff[2];
    { const int g = fq, q = (lane & 15) >> 2, p = lane & 3, row0 = 8 * g + q;
#pragma unroll
      for (int n = 0; n < 2; ++n) { const int c = 2 * wc + n; boff[n] = (int)(unsigned)(size_t)lds + SB(0, 0) + 256 * row0 + 16 * ((2 * c + (p >> 1)) ^ bswz(row0)) + 8 * (p & 1); } }
#define STAGE_A(bufoff, gbase) do { _Pragma("unroll") for (int _i = 0; _i < 2; ++_i) \
        __builtin_amdgcn_global_load_lds((const unsigned*)((const char*)(gbase) + voffA[_i]), (LAS unsigned*)(lds + (bufoff) + ldsw + _i * 8192), 16, 0, 0); } while (0)
#define STAGE_B(bufoff, gbase) do { _Pragma("unroll") for (int _i = 0; _i < 2; ++_i) \
        __builtin_amdgcn_global_load_lds((const unsigned*)((const char*)(gbase) + (size_t)_i * 32 * N * 2 + voffB), (LAS unsigned*)(lds + (bufoff) + ldsw + _i * 8192), 16, 0, 0); } while (0)
#define LDA(dst, b, h) do { _Pragma("unroll") for (int m = 0; m < 4; ++m) _Pragma("unroll") for (int k = 0; k < 2; ++k) dst[m][k] = *(const LAS bf16x8*)(lds + SA(b, h) + aoff + m * 2048 + k * 1024); } while (0)
#define TRRD(dst, addr, off) asm volatile("ds_read_b64_tr_b16 %0, %1 offset:%2" : "=v"(dst) : "v"(addr), "i"(off) : "memory")
#define LDB(dst, b, h) do { _Pragma("unroll") for (int n = 0; n < 2; ++n) _Pragma("unroll") for (int k = 0; k < 2; ++k) { \
        s16x4 lo, hi; TRRD(lo, boff[n], ((b) * 2 + (h)) * HTB + k * 8192); TRRD(hi, boff[n], ((b) * 2 + (h)) * HTB + k * 8192 + 1024); \
        dst[n][k] = __builtin_shufflevector(lo, hi, 0, 1, 2, 3, 4, 5, 6, 7); } } while (0)
#define MMA(ai, bj, At, Bt) do { __builtin_amdgcn_s_setprio(1); _Pragma("unroll") for (int m = 0; m < 4; ++m) _Pragma("unroll") for (int n = 0; n < 2; ++n) _Pragma("unroll") for (int k = 0; k < 2; ++k) \
        acc[ai][bj][m][n] = __builtin_amdgcn_mfma_f32_16x16x32_bf16(Bt[n][k], At[m][k], acc[ai][bj][m][n], 0, 0, 0); __builtin_amdgcn_s_setprio(0); } while (0)
#define WAIT_V(n) asm volatile("s_waitcnt vmcnt(" #n ")" ::: "memory")
#define WAIT_L(n) asm volatile("s_waitcnt lgkmcnt(" #n ")" ::: "memory")
#define BAR __builtin_amdgcn_s_barrier()
#define SCHED __builtin_amdgcn_sched_barrier(0)
    f32x4 acc[2][2][4][2];
#pragma unroll
    for (int a = 0; a < 2; ++a)
#pragma unroll
        for (int b = 0; b < 2; ++b)
#pragma unroll
            for (int m = 0; m < 4; ++m)
#pragma unroll
                for (int n = 0; n < 2; ++n) acc[a][b][m][n] = (f32x4){0.f, 0.f, 0.f, 0.f};
    bf16x8 At[4][2], B0[2][2], B1[2][2];
    const char* cA = (const char*)A + (size_t)pm * 2 * hstepA; const char* cB = (const char*)W + (size_t)pn * 2 * hstepB;
    STAGE_B(SB(0, 0), cB); STAGE_A(SA(0, 0), cA); STAGE_B(SB(0, 1), cB + hstepB); STAGE_A(SA(0, 1), cA + hstepA);
    if (wr == 1) BAR;
    WAIT_V(4); BAR;
    STAGE_B(SB(1, 0), cB + kstepB); STAGE_A(SA(1, 0), cA + kstepA); STAGE_B(SB(1, 1), cB + hstepB + kstepB);
    WAIT_V(6); BAR;
    for (int t = 0; t < nt; t += 2) {
        const bool last = (t == nt - 2);
        const char* a1 = cA + (size_t)(t + 1) * kstepA;
        const int t2 = last ? 0 : t + 2;
        const char* a2 = cA + (size_t)t2 * kstepA; const char* b2 = cB + (size_t)t2 * kstepB;
        const char* a3 = a2 + kstepA; const char* b3 = b2 + kstepB;
        LDB(B0, 0, 0); SCHED; LDA(At, 0, 0); STAGE_A(SA(1, 1), a1 + hstepA);
        WAIT_L(8); BAR; WAIT_L(0); SCHED; MMA(0, 0, At, B0); BAR; SCHED;
        LDB(B1, 0, 1); STAGE_B(SB(0, 0), b2);
        BAR; WAIT_L(0); SCHED; MMA(0, 1, At, B1); BAR;
        LDA(At, 0, 1); STAGE_A(SA(0, 0), a2);
        BAR; WAIT_L(0); SCHED; MMA(1, 0, At, B0); BAR; SCHED;
        STAGE_B(SB(0, 1), b2 + hstepB);
        WAIT_V(6); BAR; MMA(1, 1, At, B1); BAR;
        LDB(B0, 1, 0); SCHED; LDA(At, 1, 0); STAGE_A(SA(0, 1), a2 + hstepA);
        WAIT_L(8); BAR; WAIT_L(0); SCHED; MMA(0, 0, At, B0); BAR; SCHED;
        LDB(B1, 1, 1); STAGE_B(SB(1, 0), b3);
        BAR; WAIT_L(0); SCHED; MMA(0, 1, At, B1); BAR;
        LDA(At, 1, 1); STAGE_A(SA(1, 0), a3);
        BAR; WAIT_L(0); SCHED; MMA(1, 0, At, B0); BAR; SCHED;
        STAGE_B(SB(1, 1), b3 + hstepB);
        WAIT_V(6); BAR; MMA(1, 1, At, B1); BAR;
    }
    {
        const int row0 = pm * BM + wr * 64 + fr, col0 = pn * BM + wc * 32 + 4 * fq;
#pragma unroll
        for (int ai = 0; ai < 2; ++ai)
#pragma unroll
            for (int m = 0; m < 4; ++m) { bf16_t* rowp = O + (size_t)(row0 + ai * HALF + m * 16) * N + col0;
#pragma unroll
                for (int bj = 0; bj < 2; ++bj)
#pragma unroll
                    for (int n = 0; n < 2; ++n) { const f32x4 v = acc[ai][bj][m][n];
                        u32x2 w; w.x = cvt_pk_bf16(silu_f(v[0]), silu_f(v[1])); w.y = cvt_pk_bf16(silu_f(v[2]), silu_f(v[3]));
                        *(u32x2*)(rowp + bj * HALF + n * 16) = w; } }
    }
    WAIT_V(0);
    if (wr == 0) BAR;
}


extern "C" void kernel_launch(int rank, int nranks, ncclComm_t comm,
                              hipStream_t stream,
                              void* const* d_in, const int* in_sizes, int n_in,
                              void* d_out, int out_size,
                              const int* in_dtypes, int out_dtype,
                              void* const* d_out_all,
                              void* const* d_ws_all, size_t ws_bytes) {
    static bool init = false;
    if (!init) { (void)hipFuncSetAttribute((const void*)gemm_k, hipFuncAttributeMaxDynamicSharedMemorySize, STAGE_BYTES); init = true; }
    void* ws = d_ws_all[rank];
    ncclAllGather(d_in[0], ws, (size_t)512 * 4096 * 2, ncclInt8, comm, stream);
    gemm_k<<<256, 512, STAGE_BYTES, stream>>>((const bf16_t*)ws, (const bf16_t*)d_in[1], (bf16_t*)d_out);
}
```

```cpp
#include <hip/hip_runtime.h>
#include <rccl/rccl.h>
#include <cstdio>

namespace agg {
#define LAS __attribute__((address_space(3)))
typedef unsigned short bf16_t;
typedef short bf16x8 __attribute__((ext_vector_type(8)));
typedef short s16x4 __attribute__((ext_vector_type(4)));
typedef float f32x4 __attribute__((ext_vector_type(4)));
typedef unsigned u32x4 __attribute__((ext_vector_type(4)));
typedef unsigned u32x2 __attribute__((ext_vector_type(2)));

constexpr int NR = 8;
constexpr int MM = 4096, NN = 4096, KK = 4096, MSH = MM / NR;
constexpr int BM = 256, BK = 64, HALF = 128, HTB = HALF * BK * 2, STAGE_BYTES = 8 * HTB, NXCD = 8, WGM = 8;
constexpr int NWG = 256;
constexpr int NCH = 32;
#define AGG_D 3
constexpr int DLA = AGG_D;
constexpr size_t GATHER_BYTES = (size_t)MM * KK * 2;
constexpr size_t FLAG_OFF = 2 * GATHER_BYTES;
constexpr size_t CTL_OFF = FLAG_OFF + 64 * 1024;
constexpr size_t WS_NEED = CTL_OFF + 4096;
constexpr unsigned long long DEADLINE_TICKS = 50000000ull;

struct Params { const bf16_t* x; const bf16_t* w; bf16_t* out; unsigned char* ws[NR]; int rank; };

__host__ __device__ __forceinline__ int lds_byte(int r, int c) { const int st = (r >> 4) * 2 + (c >> 5), rr = r & 15, cc = c & 31, ob = rr * 64 + cc * 2; return st * 1024 + (ob ^ (((ob >> 9) & 1) << 5)); }
__host__ __device__ __forceinline__ void stage_rc(int b, int& R, int& C) { const int st = b / 1024, sb = b % 1024, swz = sb ^ (((sb >> 9) & 1) << 5); R = (st >> 1) * 16 + swz / 64; C = (st & 1) * 32 + (swz % 64) / 2; }
__host__ __device__ __forceinline__ int bswz(int row) { return ((row & 3) | (((row >> 3) & 1) << 2)) << 1; }

__device__ __forceinline__ unsigned cvt_pk_bf16(float lo, float hi) { unsigned r; asm volatile("v_cvt_pk_bf16_f32 %0, %1, %2" : "=v"(r) : "v"(lo), "v"(hi)); return r; }
__device__ __forceinline__ float silu_f(float y) {
    const float c = fminf(fmaxf(y, -60.f), 60.f);
    const float e = __builtin_amdgcn_exp2f(c * -1.44269504088896340736f);
    return y * __builtin_amdgcn_rcpf(1.0f + e);
}

__global__ __launch_bounds__(512, 2) void ag_gemm_k(const Params P) {
    extern __shared__ __attribute__((aligned(16))) unsigned char shm[];
    LAS unsigned char* lds = (LAS unsigned char*)shm;
    const int tid = threadIdx.x, wid = __builtin_amdgcn_readfirstlane(tid >> 6), lane = tid & 63, wr = wid >> 2, wc = wid & 3, fr = lane & 15, fq = lane >> 4;
    constexpr int K = KK, N = NN, nt = K / BK, nN = NN / BM;
    int wgid = blockIdx.x; { const int q = NWG / NXCD, xcd = wgid % NXCD, off = wgid / NXCD; wgid = xcd * q + off; }
    const int nig = WGM * nN, gid = wgid / nig, fm = gid * WGM;
    const int pm = fm + ((wgid % nig) % WGM), pn = (wgid % nig) / WGM;
    const int rank = P.rank, src = pm >> 1;
    const bool remote = (src != rank);
    unsigned char* const myws = P.ws[rank];

#define SA(b, h) (((b) * 2 + (h)) * HTB)
#define SB(b, h) ((4 + (b) * 2 + (h)) * HTB)
    unsigned voffA[2], voffB;
#pragma unroll
    for (int i = 0; i < 2; ++i) { int R, C; stage_rc(tid * 16 + i * 8192, R, C); voffA[i] = (unsigned)(R * K + C) * 2u; }
    { const int b = tid * 16, row = b >> 8, chp = (b >> 4) & 15, ch = chp ^ bswz(row); voffB = (unsigned)(row * N * 2 + ch * 16); }
    const size_t kstepA = (size_t)(BK * 2), kstepB = (size_t)BK * N * 2;
    const size_t hstepA = (size_t)HALF * K * 2, hstepB = (size_t)HALF * 2;
    const unsigned ldsw = (unsigned)wid * 1024u;
    const int aoff = lds_byte(wr * 64 + fr, fq * 8);
    int boff[2];
    { const int g = fq, q = (lane & 15) >> 2, p = lane & 3, row0 = 8 * g + q;
#pragma unroll
      for (int n = 0; n < 2; ++n) { const int c = 2 * wc + n; boff[n] = (int)(unsigned)(size_t)lds + SB(0, 0) + 256 * row0 + 16 * ((2 * c + (p >> 1)) ^ bswz(row0)) + 8 * (p & 1); } }
#define STAGE_A(bufoff, gbase) do { _Pragma("unroll") for (int _i = 0; _i < 2; ++_i) \
        __builtin_amdgcn_global_load_lds((const unsigned*)((const char*)(gbase) + voffA[_i]), (LAS unsigned*)(lds + (bufoff) + ldsw + _i * 8192), 16, 0, 0); } while (0)
#define STAGE_B(bufoff, gbase) do { _Pragma("unroll") for (int _i = 0; _i < 2; ++_i) \
        __builtin_amdgcn_global_load_lds((const unsigned*)((const char*)(gbase) + (size_t)_i * 32 * N * 2 + voffB), (LAS unsigned*)(lds + (bufoff) + ldsw + _i * 8192), 16, 0, 0); } while (0)
#define LDA(dst, b, h) do { _Pragma("unroll") for (int m = 0; m < 4; ++m) _Pragma("unroll") for (int k = 0; k < 2; ++k) dst[m][k] = *(const LAS bf16x8*)(lds + SA(b, h) + aoff + m * 2048 + k * 1024); } while (0)
#define TRRD(dst, addr, off) asm volatile("ds_read_b64_tr_b16 %0, %1 offset:%2" : "=v"(dst) : "v"(addr), "i"(off) : "memory")
#define LDB(dst, b, h) do { _Pragma("unroll") for (int n = 0; n < 2; ++n) _Pragma("unroll") for (int k = 0; k < 2; ++k) { \
        s16x4 lo, hi; TRRD(lo, boff[n], ((b) * 2 + (h)) * HTB + k * 8192); TRRD(hi, boff[n], ((b) * 2 + (h)) * HTB + k * 8192 + 1024); \
        dst[n][k] = __builtin_shufflevector(lo, hi, 0, 1, 2, 3, 4, 5, 6, 7); } } while (0)
#define MMA(ai, bj, At, Bt) do { __builtin_amdgcn_s_setprio(1); _Pragma("unroll") for (int m = 0; m < 4; ++m) _Pragma("unroll") for (int n = 0; n < 2; ++n) _Pragma("unroll") for (int k = 0; k < 2; ++k) \
        acc[ai][bj][m][n] = __builtin_amdgcn_mfma_f32_16x16x32_bf16(Bt[n][k], At[m][k], acc[ai][bj][m][n], 0, 0, 0); __builtin_amdgcn_s_setprio(0); } while (0)
#define WAIT_V(n) asm volatile("s_waitcnt vmcnt(" #n ")" ::: "memory")
#define WAIT_L(n) asm volatile("s_waitcnt lgkmcnt(" #n ")" ::: "memory")
#define BAR __builtin_amdgcn_s_barrier()
#define SCHED __builtin_amdgcn_sched_barrier(0)

    f32x4 acc[2][2][4][2];
#pragma unroll
    for (int a = 0; a < 2; ++a)
#pragma unroll
        for (int b = 0; b < 2; ++b)
#pragma unroll
            for (int m = 0; m < 4; ++m)
#pragma unroll
                for (int n = 0; n < 2; ++n) acc[a][b][m][n] = (f32x4){0.f, 0.f, 0.f, 0.f};
    bf16x8 At[4][2], B0[2][2], B1[2][2];
    const char* cB = (const char*)P.w + (size_t)pn * 2 * hstepB;

    const bool pusher = wid < NR - 1;
    const int peer = (rank + 1 + wid) & (NR - 1);
    const int j = blockIdx.x;
    const unsigned xoff = (unsigned)((2 * j + ((lane >> 4) & 1)) * K * 2 + (lane & 15) * 16);
    const char* xsrc = (const char*)P.x;
    unsigned* const ctr = (unsigned*)(myws + CTL_OFF);
    unsigned* const tmo = (unsigned*)(myws + CTL_OFF + 256);

    STAGE_B(SB(0, 0), cB); STAGE_B(SB(0, 1), cB + hstepB); STAGE_B(SB(1, 0), cB + kstepB); STAGE_B(SB(1, 1), cB + hstepB + kstepB);
    u32x4 xd[DLA + 1];
    unsigned cv;
    asm volatile("global_load_dword %0, %1, %2 sc1" : "=v"(cv) : "v"(0u), "s"(ctr) : "memory");
#pragma unroll
    for (int c = 0; c <= DLA; ++c) asm volatile("global_load_dwordx4 %0, %1, %2" : "=v"(xd[c]) : "v"(xoff), "s"(xsrc + c * 256) : "memory");
    WAIT_V(0);
    asm volatile("" : "+v"(cv));
#pragma unroll
    for (int c = 0; c <= DLA; ++c) asm volatile("" : "+v"(xd[c]));
    const unsigned call = (unsigned)__builtin_amdgcn_readfirstlane(cv) / NWG + 1u;
    const unsigned fbase = call * 64u;
    const size_t par_off = (size_t)(call & 1u) * GATHER_BYTES;
    const char* pdst = (const char*)P.ws[peer] + par_off + (size_t)rank * MSH * K * 2;
    const char* pflag = (const char*)P.ws[peer] + FLAG_OFF + (size_t)rank * 1024 + (size_t)j * 4;
    const char* qflag = (const char*)myws + FLAG_OFF + (size_t)src * 1024 + (size_t)(pm & 1) * 512;
    const unsigned qoff = (unsigned)lane * 8u;
    bool bad = false;

#define PUSH_S(data, c) asm volatile("global_store_dwordx4 %0, %1, %2 sc0 sc1\n\ts_nop 1" :: "v"(xoff), "v"(data), "s"(pdst + (size_t)(c) * 256) : "memory")
#define PUSH_F(cnt) do { const unsigned _fv = fbase + (unsigned)(cnt); asm volatile("global_store_dword %0, %1, %2 sc0 sc1" :: "v"(0u), "v"(_fv), "s"(pflag) : "memory"); } while (0)
#define PUSH_L(dst, c) asm volatile("global_load_dwordx4 %0, %1, %2" : "=v"(dst) : "v"(xoff), "s"(xsrc + (size_t)(c) * 256) : "memory")
#define POLL(dst) asm volatile("global_load_dwordx2 %0, %1, %2 sc0 sc1" : "=v"(dst) : "v"(qoff), "s"(qflag) : "memory")
#define SPIN(need) do { const unsigned _nd = fbase + (unsigned)(need); unsigned long long _t0 = __builtin_amdgcn_s_memrealtime(); \
        for (;;) { u32x2 _pf; POLL(_pf); WAIT_V(0); asm volatile("" : "+v"(_pf)); \
            if (__all(_pf.x >= _nd && _pf.y >= _nd)) break; \
            unsigned _tv; asm volatile("global_load_dword %0, %1, %2 sc1\n\ts_waitcnt vmcnt(0)" : "=v"(_tv) : "v"(0u), "s"(tmo) : "memory"); \
            if ((unsigned)__builtin_amdgcn_readfirstlane(_tv) == call || __builtin_amdgcn_s_memrealtime() - _t0 > DEADLINE_TICKS) { \
                if (lane == 0) asm volatile("global_store_dword %0, %1, %2 sc1" :: "v"(0u), "v"(call), "s"(tmo) : "memory"); break; } \
            __builtin_amdgcn_s_sleep(2); } } while (0)

    if (pusher) {
        if (lane < 32) {
#pragma unroll
            for (int c = 0; c < DLA; ++c) PUSH_S(xd[c], c);
            asm volatile("s_waitcnt vmcnt(%0)" :: "i"(DLA - 1) : "memory");
            if (lane == 0) PUSH_F(1);
            WAIT_V(0);
            if (lane == 0) PUSH_F(DLA);
        }
    } else if (remote) {
        SPIN(2);
    }
    u32x4 xcur = xd[DLA];
    const char* cA = remote ? (const char*)myws + par_off + (size_t)pm * 2 * hstepA : (const char*)P.x + (size_t)(pm & 1) * 2 * hstepA;
    BAR;
    STAGE_A(SA(0, 0), cA); STAGE_A(SA(0, 1), cA + hstepA); STAGE_A(SA(1, 0), cA + kstepA);
    WAIT_V(2); BAR;
    if (wr == 1) BAR;
    bool ready = true;
    for (int t = 0; t < nt; t += 2) {
        const int it = t >> 1;
        const bool last = (t == nt - 2);
        const char* a1 = cA + (size_t)(t + 1) * kstepA;
        const int t2 = last ? 0 : t + 2;
        const char* a2 = cA + (size_t)t2 * kstepA; const char* b2 = cB + (size_t)t2 * kstepB;
        const char* a3 = a2 + kstepA; const char* b3 = b2 + kstepB;
        const bool push_now = pusher && (it + DLA < NCH);
        LDB(B0, 0, 0); SCHED; LDA(At, 0, 0); STAGE_A(SA(1, 1), a1 + hstepA);
        if (push_now) {
            if (lane < 32) {
                if (lane == 0) PUSH_F(it + DLA);
                PUSH_S(xcur, it + DLA);
                PUSH_L(xcur, (it + DLA + 1) & (NCH - 1));
            }
        } else if (pusher && it + DLA == NCH) {
            if (lane == 0) PUSH_F(NCH);
        } else if (!pusher && !ready) {
            SPIN(it + 2);
        }
        WAIT_L(8); BAR; WAIT_L(0); SCHED; MMA(0, 0, At, B0); BAR; SCHED;
        LDB(B1, 0, 1); STAGE_B(SB(0, 0), b2);
        BAR; WAIT_L(0); SCHED; MMA(0, 1, At, B1); BAR;
        LDA(At, 0, 1); STAGE_A(SA(0, 0), a2);
        BAR; WAIT_L(0); SCHED; MMA(1, 0, At, B0); BAR; SCHED;
        STAGE_B(SB(0, 1), b2 + hstepB);
        if (push_now) WAIT_V(9); else WAIT_V(6);
        BAR; MMA(1, 1, At, B1); BAR;
        LDB(B0, 1, 0); SCHED; LDA(At, 1, 0); STAGE_A(SA(0, 1), a2 + hstepA);
        WAIT_L(8); BAR; WAIT_L(0); SCHED; MMA(0, 0, At, B0); BAR; SCHED;
        u32x2 pf = (u32x2){0xffffffffu, 0xffffffffu};
        const bool do_poll = !pusher && remote && (it + 2 < NCH);
        if (do_poll) POLL(pf);
        LDB(B1, 1, 1); STAGE_B(SB(1, 0), b3);
        BAR; WAIT_L(0); SCHED; MMA(0, 1, At, B1); BAR;
        LDA(At, 1, 1); STAGE_A(SA(1, 0), a3);
        BAR; WAIT_L(0); SCHED; MMA(1, 0, At, B0); BAR; SCHED;
        STAGE_B(SB(1, 1), b3 + hstepB);
        WAIT_V(6);
        asm volatile("" : "+v"(pf), "+v"(xcur));
        { const unsigned nd = fbase + (unsigned)(it + 3); ready = __all(pf.x >= nd && pf.y >= nd); }
        BAR; MMA(1, 1, At, B1); BAR;
    }
    {
        unsigned tv; asm volatile("global_load_dword %0, %1, %2 sc1\n\ts_waitcnt vmcnt(0)" : "=v"(tv) : "v"(0u), "s"(tmo) : "memory");
        bad = ((unsigned)__builtin_amdgcn_readfirstlane(tv) == call);
        const int row0 = pm * BM + wr * 64 + fr, col0 = pn * BM + wc * 32 + 4 * fq;
#pragma unroll
        for (int ai = 0; ai < 2; ++ai)
#pragma unroll
            for (int m = 0; m < 4; ++m) { bf16_t* rowp = P.out + (size_t)(row0 + ai * HALF + m * 16) * N + col0;
#pragma unroll
                for (int bj = 0; bj < 2; ++bj)
#pragma unroll
                    for (int n = 0; n < 2; ++n) { const f32x4 v = acc[ai][bj][m][n];
                        u32x2 w; w.x = cvt_pk_bf16(silu_f(v[0]), silu_f(v[1])); w.y = cvt_pk_bf16(silu_f(v[2]), silu_f(v[3]));
                        if (bad) { w.x = 0x7fc07fc0u; w.y = 0x7fc07fc0u; }
                        *(u32x2*)(rowp + bj * HALF + n * 16) = w; } }
    }
    if (tid == 0) __hip_atomic_fetch_add(ctr, 1u, __ATOMIC_RELAXED, __HIP_MEMORY_SCOPE_AGENT);
    WAIT_V(0);
    if (wr == 0) BAR;
}
}

extern "C" void kernel_launch(int rank, int nranks, ncclComm_t comm,
                              hipStream_t stream,
                              void* const* d_in, const int* in_sizes, int n_in,
                              void* d_out, int out_size,
                              const int* in_dtypes, int out_dtype,
                              void* const* d_out_all,
                              void* const* d_ws_all, size_t ws_bytes) {
    using namespace agg;
    static bool init = false;
    if (!init) { (void)hipFuncSetAttribute((const void*)ag_gemm_k, hipFuncAttributeMaxDynamicSharedMemorySize, STAGE_BYTES); init = true; }
    if (nranks != NR || ws_bytes < WS_NEED) { fprintf(stderr, "unsupported configuration\n"); return; }
    Params p;
    p.x = (const bf16_t*)d_in[0]; p.w = (const bf16_t*)d_in[1]; p.out = (bf16_t*)d_out; p.rank = rank;
    for (int i = 0; i < NR; ++i) p.ws[i] = (unsigned char*)d_ws_all[i];
    ag_gemm_k<<<NWG, 512, STAGE_BYTES, stream>>>(p);
}
```

```cpp
#include <hip/hip_runtime.h>
#include <rccl/rccl.h>
#include <cstdio>

namespace agg {
#define LAS __attribute__((address_space(3)))
typedef unsigned short bf16_t;
typedef short bf16x8 __attribute__((ext_vector_type(8)));
typedef short s16x4 __attribute__((ext_vector_type(4)));
typedef float f32x4 __attribute__((ext_vector_type(4)));
typedef unsigned u32x4 __attribute__((ext_vector_type(4)));
typedef unsigned u32x2 __attribute__((ext_vector_type(2)));

constexpr int NR = 8;
constexpr int MM = 4096, NN = 4096, KK = 4096, MSH = MM / NR;
constexpr int BM = 256, BK = 64, HALF = 128, HTB = HALF * BK * 2, GEMM_LDS = 8 * HTB, NXCD = 8, WGM = 8;
constexpr int XBUF = GEMM_LDS, PBUF = GEMM_LDS + 1024, STAGE_BYTES = GEMM_LDS + 2048;
constexpr int NWG = 256;
constexpr int NCH = 32;
#define AGG_D 4
constexpr int DLA = AGG_D;
constexpr size_t GATHER_BYTES = (size_t)MM * KK * 2;
constexpr size_t FLAG_OFF = 2 * GATHER_BYTES;
constexpr size_t CTL_OFF = FLAG_OFF + 64 * 1024;
constexpr size_t WS_NEED = CTL_OFF + 4096;
constexpr unsigned long long DEADLINE_TICKS = 50000000ull;

struct Params { const bf16_t* x; const bf16_t* w; bf16_t* out; unsigned char* ws[NR]; int rank; };

__host__ __device__ __forceinline__ int lds_byte(int r, int c) { const int st = (r >> 4) * 2 + (c >> 5), rr = r & 15, cc = c & 31, ob = rr * 64 + cc * 2; return st * 1024 + (ob ^ (((ob >> 9) & 1) << 5)); }
__host__ __device__ __forceinline__ void stage_rc(int b, int& R, int& C) { const int st = b / 1024, sb = b % 1024, swz = sb ^ (((sb >> 9) & 1) << 5); R = (st >> 1) * 16 + swz / 64; C = (st & 1) * 32 + (swz % 64) / 2; }
__host__ __device__ __forceinline__ int bswz(int row) { return ((row & 3) | (((row >> 3) & 1) << 2)) << 1; }

__device__ __forceinline__ unsigned cvt_pk_bf16(float lo, float hi) { unsigned r; asm volatile("v_cvt_pk_bf16_f32 %0, %1, %2" : "=v"(r) : "v"(lo), "v"(hi)); return r; }
__device__ __forceinline__ float silu_f(float y) {
    const float c = fminf(fmaxf(y, -60.f), 60.f);
    const float e = __builtin_amdgcn_exp2f(c * -1.44269504088896340736f);
    return y * __builtin_amdgcn_rcpf(1.0f + e);
}

__global__ __launch_bounds__(512, 2) void ag_gemm_k(const Params P) {
    extern __shared__ __attribute__((aligned(16))) unsigned char shm[];
    LAS unsigned char* lds = (LAS unsigned char*)shm;
    const int tid = threadIdx.x, wid = __builtin_amdgcn_readfirstlane(tid >> 6), lane = tid & 63, wr = wid >> 2, wc = wid & 3, fr = lane & 15, fq = lane >> 4;
    constexpr int K = KK, N = NN, nt = K / BK, nN = NN / BM;
    int wgid = blockIdx.x; { const int q = NWG / NXCD, xcd = wgid % NXCD, off = wgid / NXCD; wgid = xcd * q + off; }
    const int nig = WGM * nN, gid = wgid / nig, fm = gid * WGM;
    const int pm = fm + ((wgid % nig) % WGM), pn = (wgid % nig) / WGM;
    const int rank = P.rank, src = pm >> 1;
    const bool remote = (src != rank);
    unsigned char* const myws = P.ws[rank];
    const bool comm = (wid == 7);
#define AGG_BDELAY 1
#define AGG_CAP 4
#define AGG_W 1
#define AGG_H2 6
#define AGG_HELP_IN_M 1
    const bool helper = (wid == 3 || wid == AGG_H2);

#define SA(b, h) (((b) * 2 + (h)) * HTB)
#define SB(b, h) ((4 + (b) * 2 + (h)) * HTB)
    unsigned voffA[2], voffB;
#pragma unroll
    for (int i = 0; i < 2; ++i) { int R, C; stage_rc(tid * 16 + i * 8192, R, C); voffA[i] = (unsigned)(R * K + C) * 2u; }
    { const int b = tid * 16, row = b >> 8, chp = (b >> 4) & 15, ch = chp ^ bswz(row); voffB = (unsigned)(row * N * 2 + ch * 16); }
    const size_t kstepA = (size_t)(BK * 2), kstepB = (size_t)BK * N * 2;
    const size_t hstepA = (size_t)HALF * K * 2, hstepB = (size_t)HALF * 2;
    const unsigned ldsw = (unsigned)wid * 1024u;
    const unsigned hA = (wid == 3) ? 32u * K * 2u : (7u - (AGG_H2 >> 1)) * 16u * K * 2u + 64u;
    const unsigned hB = (wid == 3) ? 16u * N * 2u : (15u - AGG_H2) * 4u * N * 2u;
    const unsigned hL = (wid == 3) ? 7u * 1024u : 15u * 1024u;
    const int aoff = lds_byte(wr * 64 + fr, fq * 8);
    int boff[2];
    { const int g = fq, q = (lane & 15) >> 2, p = lane & 3, row0 = 8 * g + q;
#pragma unroll
      for (int n = 0; n < 2; ++n) { const int c = 2 * wc + n; boff[n] = (int)(unsigned)(size_t)lds + SB(0, 0) + 256 * row0 + 16 * ((2 * c + (p >> 1)) ^ bswz(row0)) + 8 * (p & 1); } }
#define GLDS(g, l) __builtin_amdgcn_global_load_lds((const unsigned*)(g), (LAS unsigned*)(l), 16, 0, 0)
#define STAGE_A(bufoff, gbase) do { if (!comm) { _Pragma("unroll") for (int _i = 0; _i < 2; ++_i) GLDS((const char*)(gbase) + voffA[_i], lds + (bufoff) + ldsw + _i * 8192); } } while (0)
#define STAGE_B(bufoff, gbase) do { if (!comm) { _Pragma("unroll") for (int _i = 0; _i < 2; ++_i) GLDS((const char*)(gbase) + (size_t)_i * 32 * N * 2 + voffB, lds + (bufoff) + ldsw + _i * 8192); } } while (0)
#define HELP_A(bufoff, gbase) do { if (helper) GLDS((const char*)(gbase) + hA + voffA[0], lds + (bufoff) + hL); } while (0)
#define HELP_B(bufoff, gbase) do { if (helper) GLDS((const char*)(gbase) + hB + voffB, lds + (bufoff) + hL); } while (0)
#define LDA(dst, b, h) do { _Pragma("unroll") for (int m = 0; m < 4; ++m) _Pragma("unroll") for (int k = 0; k < 2; ++k) dst[m][k] = *(const LAS bf16x8*)(lds + SA(b, h) + aoff + m * 2048 + k * 1024); } while (0)
#define TRRD(dst, addr, off) asm volatile("ds_read_b64_tr_b16 %0, %1 offset:%2" : "=v"(dst) : "v"(addr), "i"(off) : "memory")
#define LDB(dst, b, h) do { _Pragma("unroll") for (int n = 0; n < 2; ++n) _Pragma("unroll") for (int k = 0; k < 2; ++k) { \
        s16x4 lo, hi; TRRD(lo, boff[n], ((b) * 2 + (h)) * HTB + k * 8192); TRRD(hi, boff[n], ((b) * 2 + (h)) * HTB + k * 8192 + 1024); \
        dst[n][k] = __builtin_shufflevector(lo, hi, 0, 1, 2, 3, 4, 5, 6, 7); } } while (0)
#define MMA_K(ai, bj, At, Bt, k) do { _Pragma("unroll") for (int m = 0; m < 4; ++m) _Pragma("unroll") for (int n = 0; n < 2; ++n) \
        acc[ai][bj][m][n] = __builtin_amdgcn_mfma_f32_16x16x32_bf16(Bt[n][k], At[m][k], acc[ai][bj][m][n], 0, 0, 0); } while (0)
#define MMA(ai, bj, At, Bt, MID) do { __builtin_amdgcn_s_setprio(1); MMA_K(ai, bj, At, Bt, 0); SCHED; MID; SCHED; MMA_K(ai, bj, At, Bt, 1); __builtin_amdgcn_s_setprio(0); } while (0)
#define WAIT_V(n) asm volatile("s_waitcnt vmcnt(" #n ")" ::: "memory")
#define WAIT_V2(n) WAIT_V(n)
#define WAIT_VC(n2, n3) do { if (!comm) { if (helper) WAIT_V2(n3); else WAIT_V2(n2); } } while (0)
#define HR(x) do {} while (0)
#define HM(x) x
#define HWAIT 8
#define WAIT_L(n) asm volatile("s_waitcnt lgkmcnt(" #n ")" ::: "memory")
#define BAR __builtin_amdgcn_s_barrier()
#define SCHED __builtin_amdgcn_sched_barrier(0)

#define STAMP() do {} while (0)
#define STAMP2(i) do {} while (0)
    STAMP();
    f32x4 acc[2][2][4][2];
#pragma unroll
    for (int a = 0; a < 2; ++a)
#pragma unroll
        for (int b = 0; b < 2; ++b)
#pragma unroll
            for (int m = 0; m < 4; ++m)
#pragma unroll
                for (int n = 0; n < 2; ++n) acc[a][b][m][n] = (f32x4){0.f, 0.f, 0.f, 0.f};
    bf16x8 At[4][2], B0[2][2], B1[2][2];
    const char* cB = (const char*)P.w + (size_t)pn * 2 * hstepB;

    const int j = blockIdx.x;
    unsigned xo = (unsigned)((2 * j + ((lane >> 4) & 1)) * K * 2 + (lane & 15) * 16);
    const char* xsrc = (const char*)P.x;
    unsigned* const ctr = (unsigned*)(myws + CTL_OFF);
    unsigned* const tmo = (unsigned*)(myws + CTL_OFF + 256);

    unsigned cv;
    asm volatile("global_load_dword %0, %1, %2 sc1" : "=v"(cv) : "v"(0u), "s"(ctr) : "memory");
    if (comm && lane < 32) __builtin_amdgcn_global_load_lds((const unsigned*)(xsrc + xo), (LAS unsigned*)(lds + XBUF), 16, 0, 0);
    WAIT_V(0);
    asm volatile("" : "+v"(cv));
    STAMP();
    const unsigned call = (unsigned)__builtin_amdgcn_readfirstlane(cv) / NWG + 1u;
    const unsigned fbase = call * 64u;
    const size_t par_off = (size_t)(call & 1u) * GATHER_BYTES;
    const size_t slot_off = par_off + (size_t)rank * MSH * K * 2;
    unsigned pdlo, pdhi;
    { const char* b = (const char*)P.ws[(rank + 1) & (NR - 1)];
#pragma unroll
      for (int i = 1; i < NR - 1; ++i) b = (lane == i) ? (const char*)P.ws[(rank + 1 + i) & (NR - 1)] : b;
      const unsigned long long u = (unsigned long long)(b + slot_off); pdlo = (unsigned)u; pdhi = (unsigned)(u >> 32); }
    u32x4 xv = (u32x4){0u, 0u, 0u, 0u};
    int e_k = 0, e_pk = (int)(blockIdx.x % (NR - 1));
    const char* pflag;
    { const char* b = (const char*)P.ws[(rank + 1) & (NR - 1)];
#pragma unroll
      for (int i = 1; i < NR - 1; ++i) b = (lane == i) ? (const char*)P.ws[(rank + 1 + i) & (NR - 1)] : b;
      pflag = b + FLAG_OFF + (size_t)rank * 1024 + (size_t)j * 4; }
    const int qsrc = remote ? src : ((rank + 1) & (NR - 1));
    const char* qflag = (const char*)myws + FLAG_OFF + (size_t)qsrc * 1024 + (size_t)(pm & 1) * 512;
    const unsigned qoff = (unsigned)lane * 16u;
    const unsigned lds0 = (unsigned)(size_t)lds;

    unsigned e_issued = 0, e_idxL = 0, e_sOld = 0, e_sNew = 0, e_idxP = 0;
    int e_nfl = 0, e_r = 0, e_acked = 0, e_have = 0; bool e_pp = false;
    bool timed_out = false;
#define VMCNT_NOW(dst) do { unsigned _v; asm volatile("s_getreg_b32 %0, hwreg(HW_REG_IB_STS, 0, 24)" : "=s"(_v) :: "memory"); dst = (_v & 15u) | ((_v >> 18) & 48u); } while (0)
#define TICK(want, lim) do { unsigned _c; VMCNT_NOW(_c); const unsigned _done = e_issued - _c; \
        if (e_nfl > 0 && (int)(_done - e_sOld) >= 0) { STAMP2(e_acked); ++e_acked; --e_nfl; e_sOld = e_sNew; \
            if (lane < NR - 1) { const unsigned _fv = fbase + (unsigned)e_acked; asm volatile("global_store_dword %0, %1, off sc0 sc1" :: "v"(pflag), "v"(_fv) : "memory"); } ++e_issued; } \
        if (_c < (e_r == 0 ? 12u : (unsigned)AGG_CAP) && (e_k > 0 || (e_r < (lim) && (e_r == 0 || e_have >= 1) && e_nfl < 2 && (int)(_done - e_idxL) >= 0))) { \
            if (e_k == 0) { asm volatile("ds_read_b128 %0, %1\n\ts_waitcnt lgkmcnt(0)" : "=&v"(xv) : "v"(lds0 + XBUF + qoff) : "memory"); \
                if (e_r + 1 < NCH) { if (lane < 32) __builtin_amdgcn_global_load_lds((const unsigned*)(xsrc + xo + 256u), (LAS unsigned*)(lds + XBUF), 16, 0, 0); ++e_issued; e_idxL = e_issued; } } \
            { const unsigned long long _b = (unsigned long long)(unsigned)__builtin_amdgcn_readlane(pdlo, e_pk) | ((unsigned long long)(unsigned)__builtin_amdgcn_readlane(pdhi, e_pk) << 32); \
              if (lane < 32) asm volatile("global_store_dwordx4 %0, %1, %2 sc0 sc1\n\ts_nop 1" :: "v"(xo), "v"(xv), "s"(_b) : "memory"); } \
            ++e_issued; ++e_k; e_pk = (e_pk == NR - 2) ? 0 : e_pk + 1; \
            if (e_k == NR - 1) { e_k = 0; if (e_nfl == 0) e_sOld = e_issued; e_sNew = e_issued; ++e_nfl; ++e_r; xo += 256u; } } \
        if (e_pp && (int)(_done - e_idxP) >= 0) { e_pp = false; u32x2 _pf; \
            asm volatile("ds_read_b64 %0, %1\n\ts_waitcnt lgkmcnt(0)" : "=&v"(_pf) : "v"(lds0 + PBUF + (unsigned)lane * 8u) : "memory"); \
            const unsigned _n1 = fbase + (unsigned)e_have + 1u; \
            if (__all(_pf.x >= _n1 && _pf.y >= _n1)) { STAMP2(32 + e_have); ++e_have; if (__all(_pf.x > _n1 && _pf.y > _n1)) { STAMP2(32 + e_have); ++e_have; } } } \
        if (!e_pp && e_have < (want)) { if (lane < 32) __builtin_amdgcn_global_load_lds((const unsigned*)(qflag + qoff), (LAS unsigned*)(lds + PBUF), 16, 0, 17); ++e_issued; e_idxP = e_issued; e_pp = true; } \
    } while (0)
#define ENGINE_UNTIL(cond, want, lim) do { if (!(cond)) { const unsigned long long _t0 = __builtin_amdgcn_s_memrealtime(); unsigned _sp = 0; \
        while (!(cond)) { TICK(want, lim); \
            if ((++_sp & 63u) == 0u) { if (__builtin_amdgcn_s_memrealtime() - _t0 > DEADLINE_TICKS) { timed_out = true; break; } } \
            __builtin_amdgcn_s_sleep(1); } \
        if (timed_out) { if (lane == 0) asm volatile("global_store_dword %0, %1, %2 sc1" :: "v"(0u), "v"(call), "s"(tmo) : "memory"); ++e_issued; e_have = NCH + 2; } } } while (0)

    if (comm) { TICK(remote ? 2 : 1, DLA); }
    BAR;
    STAGE_B(SB(0, 0), cB); HELP_B(SB(0, 0), cB); STAGE_B(SB(0, 1), cB + hstepB); HELP_B(SB(0, 1), cB + hstepB);
    STAGE_B(SB(1, 0), cB + kstepB); HELP_B(SB(1, 0), cB + kstepB); STAGE_B(SB(1, 1), cB + hstepB + kstepB); HELP_B(SB(1, 1), cB + hstepB + kstepB);
    if (comm && remote) { ENGINE_UNTIL(e_have >= 1, 2, DLA); }
    STAMP();
    const char* cA = remote ? (const char*)myws + par_off + (size_t)pm * 2 * hstepA : (const char*)P.x + (size_t)(pm & 1) * 2 * hstepA;
    BAR;
    STAGE_A(SA(0, 0), cA); HELP_A(SA(0, 0), cA); STAGE_A(SA(0, 1), cA + hstepA); HELP_A(SA(0, 1), cA + hstepA); STAGE_A(SA(1, 0), cA + kstepA); HELP_A(SA(1, 0), cA + kstepA);
    if (comm) TICK(remote ? 2 : 1, DLA);
    WAIT_VC(2, 3); BAR;
    if (wr == 1) BAR;
    STAMP();
    for (int t = 0; t < nt; t += 2) {
        const int it = t >> 1;
        const bool last = (t == nt - 2);
        const char* a1 = cA + (size_t)(t + 1) * kstepA;
        const int t2 = last ? 0 : t + 2;
        const char* a2 = cA + (size_t)t2 * kstepA; const char* b2 = cB + (size_t)t2 * kstepB;
        const char* a3 = a2 + kstepA; const char* b3 = b2 + kstepB;
        const int want = remote ? ((it + 3 < NCH) ? it + 3 : NCH) : 1;
        const int need = remote ? ((it + 2 < NCH) ? it + 2 : NCH) : 0;
        const int lim = (it + DLA < NCH) ? it + DLA : NCH;
        LDB(B0, 0, 0); SCHED; LDA(At, 0, 0); STAGE_A(SA(1, 1), a1 + hstepA); HR(HELP_A(SA(1, 1), a1 + hstepA));
        if (comm) TICK(want, lim);
        WAIT_L(8); BAR; WAIT_L(0); SCHED; MMA(0, 0, At, B0, HM(HELP_A(SA(1, 1), a1 + hstepA))); BAR; SCHED;
        LDB(B1, 0, 1); STAGE_B(SB(0, 0), b2); HR(HELP_B(SB(0, 0), b2));
        if (comm) { TICK(want, lim); ENGINE_UNTIL(e_have >= need, want, lim); }
        BAR; WAIT_L(0); SCHED; MMA(0, 1, At, B1, HM(HELP_B(SB(0, 0), b2))); BAR;
        LDA(At, 0, 1); STAGE_A(SA(0, 0), a2); HR(HELP_A(SA(0, 0), a2));
        if (comm) TICK(want, lim);
        BAR; WAIT_L(0); SCHED; MMA(1, 0, At, B0, HM(HELP_A(SA(0, 0), a2))); BAR; SCHED;
        STAGE_B(SB(0, 1), b2 + hstepB); HR(HELP_B(SB(0, 1), b2 + hstepB));
        if (comm) TICK(want, lim);
        WAIT_VC(6, HWAIT);
        BAR; MMA(1, 1, At, B1, HM(HELP_B(SB(0, 1), b2 + hstepB))); BAR;
        LDB(B0, 1, 0); SCHED; LDA(At, 1, 0); STAGE_A(SA(0, 1), a2 + hstepA); HR(HELP_A(SA(0, 1), a2 + hstepA));
        if (comm) TICK(want, lim);
        WAIT_L(8); BAR; WAIT_L(0); SCHED; MMA(0, 0, At, B0, HM(HELP_A(SA(0, 1), a2 + hstepA))); BAR; SCHED;
        LDB(B1, 1, 1); STAGE_B(SB(1, 0), b3); HR(HELP_B(SB(1, 0), b3));
        if (comm) TICK(want, lim);
        BAR; WAIT_L(0); SCHED; MMA(0, 1, At, B1, HM(HELP_B(SB(1, 0), b3))); BAR;
        LDA(At, 1, 1); STAGE_A(SA(1, 0), a3); HR(HELP_A(SA(1, 0), a3));
        if (comm) TICK(want, lim);
        BAR; WAIT_L(0); SCHED; MMA(1, 0, At, B0, HM(HELP_A(SA(1, 0), a3))); BAR; SCHED;
        STAGE_B(SB(1, 1), b3 + hstepB); HR(HELP_B(SB(1, 1), b3 + hstepB));
        if (comm) TICK(want, lim);
        WAIT_VC(6, HWAIT);
        BAR; MMA(1, 1, At, B1, HM(HELP_B(SB(1, 1), b3 + hstepB))); BAR;
        STAMP();
    }
    if (comm) { ENGINE_UNTIL(e_acked >= NCH, 1, NCH); }
    {
        unsigned tv; asm volatile("global_load_dword %0, %1, %2 sc1\n\ts_waitcnt vmcnt(0)" : "=v"(tv) : "v"(0u), "s"(tmo) : "memory");
        const bool bad = ((unsigned)__builtin_amdgcn_readfirstlane(tv) == call);
        const int row0 = pm * BM + wr * 64 + fr, col0 = pn * BM + wc * 32 + 4 * fq;
#pragma unroll
        for (int ai = 0; ai < 2; ++ai)
#pragma unroll
            for (int m = 0; m < 4; ++m) { bf16_t* rowp = P.out + (size_t)(row0 + ai * HALF + m * 16) * N + col0;
#pragma unroll
                for (int bj = 0; bj < 2; ++bj)
#pragma unroll
                    for (int n = 0; n < 2; ++n) { const f32x4 v = acc[ai][bj][m][n];
                        u32x2 w; w.x = cvt_pk_bf16(silu_f(v[0]), silu_f(v[1])); w.y = cvt_pk_bf16(silu_f(v[2]), silu_f(v[3]));
                        if (bad) { w.x = 0x7fc07fc0u; w.y = 0x7fc07fc0u; }
                        *(u32x2*)(rowp + bj * HALF + n * 16) = w; } }
    }
    if (tid == 0) __hip_atomic_fetch_add(ctr, 1u, __ATOMIC_RELAXED, __HIP_MEMORY_SCOPE_AGENT);
    WAIT_V(0);
    if (wr == 0) BAR;
}
}

extern "C" void kernel_launch(int rank, int nranks, ncclComm_t comm,
                              hipStream_t stream,
                              void* const* d_in, const int* in_sizes, int n_in,
                              void* d_out, int out_size,
                              const int* in_dtypes, int out_dtype,
                              void* const* d_out_all,
                              void* const* d_ws_all, size_t ws_bytes) {
    using namespace agg;
    static bool init = false;
    if (!init) { (void)hipFuncSetAttribute((const void*)ag_gemm_k, hipFuncAttributeMaxDynamicSharedMemorySize, STAGE_BYTES); init = true; }
    if (nranks != NR || ws_bytes < WS_NEED) { fprintf(stderr, "unsupported configuration\n"); return; }
    Params p;
    p.x = (const bf16_t*)d_in[0]; p.w = (const bf16_t*)d_in[1]; p.out = (bf16_t*)d_out; p.rank = rank;
    for (int i = 0; i < NR; ++i) p.ws[i] = (unsigned char*)d_ws_all[i];
    ag_gemm_k<<<NWG, 512, STAGE_BYTES, stream>>>(p);
}
```

```cpp
#include <hip/hip_runtime.h>
#include <rccl/rccl.h>
#include <cstdio>

namespace agg {
#define LAS __attribute__((address_space(3)))
typedef unsigned short bf16_t;
typedef short bf16x8 __attribute__((ext_vector_type(8)));
typedef short s16x4 __attribute__((ext_vector_type(4)));
typedef float f32x4 __attribute__((ext_vector_type(4)));
typedef unsigned u32x4 __attribute__((ext_vector_type(4)));
typedef unsigned u32x2 __attribute__((ext_vector_type(2)));

constexpr int NR = 8;
constexpr int MM = 4096, NN = 4096, KK = 4096, MSH = MM / NR;
constexpr int BM = 256, BK = 64, HALF = 128, HTB = HALF * BK * 2, GEMM_LDS = 8 * HTB, NXCD = 8, WGM = 8;
constexpr int XBUF = GEMM_LDS, PBUF = GEMM_LDS + 4096, STAGE_BYTES = GEMM_LDS + 4096 + 512;
constexpr int NWG = 256;
constexpr int NCH = 32;
#define AGG_D 8
constexpr int DLA = AGG_D;
constexpr size_t GATHER_BYTES = (size_t)MM * KK * 2;
constexpr size_t FLAG_OFF = 2 * GATHER_BYTES;
constexpr size_t CTL_OFF = FLAG_OFF + 64 * 1024;
constexpr size_t WS_NEED = CTL_OFF + 4096;
constexpr unsigned long long DEADLINE_TICKS = 50000000ull;

struct Params { const bf16_t* x; const bf16_t* w; bf16_t* out; unsigned char* ws[NR]; int rank; };

__host__ __device__ __forceinline__ int lds_byte(int r, int c) { const int st = (r >> 4) * 2 + (c >> 5), rr = r & 15, cc = c & 31, ob = rr * 64 + cc * 2; return st * 1024 + (ob ^ (((ob >> 9) & 1) << 5)); }
__host__ __device__ __forceinline__ void stage_rc(int b, int& R, int& C) { const int st = b / 1024, sb = b % 1024, swz = sb ^ (((sb >> 9) & 1) << 5); R = (st >> 1) * 16 + swz / 64; C = (st & 1) * 32 + (swz % 64) / 2; }
__host__ __device__ __forceinline__ int bswz(int row) { return ((row & 3) | (((row >> 3) & 1) << 2)) << 1; }

__device__ __forceinline__ unsigned cvt_pk_bf16(float lo, float hi) { unsigned r; asm volatile("v_cvt_pk_bf16_f32 %0, %1, %2" : "=v"(r) : "v"(lo), "v"(hi)); return r; }
__device__ __forceinline__ float silu_f(float y) {
    const float c = fminf(fmaxf(y, -60.f), 60.f);
    const float e = __builtin_amdgcn_exp2f(c * -1.44269504088896340736f);
    return y * __builtin_amdgcn_rcpf(1.0f + e);
}

__global__ __launch_bounds__(512, 2) void ag_gemm_k(const Params P) {
    extern __shared__ __attribute__((aligned(16))) unsigned char shm[];
    LAS unsigned char* lds = (LAS unsigned char*)shm;
    const int tid = threadIdx.x, wid = __builtin_amdgcn_readfirstlane(tid >> 6), lane = tid & 63, wr = wid >> 2, wc = wid & 3, fr = lane & 15, fq = lane >> 4;
    constexpr int K = KK, N = NN, nt = K / BK, nN = NN / BM;
    int wgid = blockIdx.x; { const int q = NWG / NXCD, xcd = wgid % NXCD, off = wgid / NXCD; wgid = xcd * q + off; }
    const int nig = WGM * nN, gid = wgid / nig, fm = gid * WGM;
    const int pm = fm + ((wgid % nig) % WGM), pn = (wgid % nig) / WGM;
    const int rank = P.rank, src = pm >> 1;
    const bool remote = (src != rank);
    unsigned char* const myws = P.ws[rank];
    const bool comm = (wid == 7);
#define AGG_H2 2
#define AGG_W1 2
    const bool helper = (wid == 3 || wid == AGG_H2);
    const bool poller = (wid == 4);

#define SA(b, h) (((b) * 2 + (h)) * HTB)
#define SB(b, h) ((4 + (b) * 2 + (h)) * HTB)
    unsigned voffA[2], voffB;
#pragma unroll
    for (int i = 0; i < 2; ++i) { int R, C; stage_rc(tid * 16 + i * 8192, R, C); voffA[i] = (unsigned)(R * K + C) * 2u; }
    { const int b = tid * 16, row = b >> 8, chp = (b >> 4) & 15, ch = chp ^ bswz(row); voffB = (unsigned)(row * N * 2 + ch * 16); }
    const size_t kstepA = (size_t)(BK * 2), kstepB = (size_t)BK * N * 2;
    const size_t hstepA = (size_t)HALF * K * 2, hstepB = (size_t)HALF * 2;
    const unsigned ldsw = (unsigned)wid * 1024u;
    const unsigned hA = (wid == 3) ? 32u * K * 2u : (7u - (AGG_H2 >> 1)) * 16u * K * 2u + ((AGG_H2 & 1) ? 0u : 64u);
    const unsigned hB = (wid == 3) ? 16u * N * 2u : (15u - AGG_H2) * 4u * N * 2u;
    const unsigned hL = (wid == 3) ? 7u * 1024u : 15u * 1024u;
    const int aoff = lds_byte(wr * 64 + fr, fq * 8);
    int boff[2];
    { const int g = fq, q = (lane & 15) >> 2, p = lane & 3, row0 = 8 * g + q;
#pragma unroll
      for (int n = 0; n < 2; ++n) { const int c = 2 * wc + n; boff[n] = (int)(unsigned)(size_t)lds + SB(0, 0) + 256 * row0 + 16 * ((2 * c + (p >> 1)) ^ bswz(row0)) + 8 * (p & 1); } }
#define GLDS(g, l) __builtin_amdgcn_global_load_lds((const unsigned*)(g), (LAS unsigned*)(l), 16, 0, 0)
#define STAGE_A(bufoff, gbase) do { if (!comm) { _Pragma("unroll") for (int _i = 0; _i < 2; ++_i) GLDS((const char*)(gbase) + voffA[_i], lds + (bufoff) + ldsw + _i * 8192); } } while (0)
#define STAGE_B(bufoff, gbase) do { if (!comm) { _Pragma("unroll") for (int _i = 0; _i < 2; ++_i) GLDS((const char*)(gbase) + (size_t)_i * 32 * N * 2 + voffB, lds + (bufoff) + ldsw + _i * 8192); } } while (0)
#define HELP_A(bufoff, gbase) do { if (helper) GLDS((const char*)(gbase) + hA + voffA[0], lds + (bufoff) + hL); } while (0)
#define HELP_B(bufoff, gbase) do { if (helper) GLDS((const char*)(gbase) + hB + voffB, lds + (bufoff) + hL); } while (0)
#define LDA(dst, b, h) do { _Pragma("unroll") for (int m = 0; m < 4; ++m) _Pragma("unroll") for (int k = 0; k < 2; ++k) dst[m][k] = *(const LAS bf16x8*)(lds + SA(b, h) + aoff + m * 2048 + k * 1024); } while (0)
#define TRRD(dst, addr, off) asm volatile("ds_read_b64_tr_b16 %0, %1 offset:%2" : "=v"(dst) : "v"(addr), "i"(off) : "memory")
#define LDB(dst, b, h) do { _Pragma("unroll") for (int n = 0; n < 2; ++n) _Pragma("unroll") for (int k = 0; k < 2; ++k) { \
        s16x4 lo, hi; TRRD(lo, boff[n], ((b) * 2 + (h)) * HTB + k * 8192); TRRD(hi, boff[n], ((b) * 2 + (h)) * HTB + k * 8192 + 1024); \
        dst[n][k] = __builtin_shufflevector(lo, hi, 0, 1, 2, 3, 4, 5, 6, 7); } } while (0)
#define MMA_K(ai, bj, At, Bt, k) do { _Pragma("unroll") for (int m = 0; m < 4; ++m) _Pragma("unroll") for (int n = 0; n < 2; ++n) \
        acc[ai][bj][m][n] = __builtin_amdgcn_mfma_f32_16x16x32_bf16(Bt[n][k], At[m][k], acc[ai][bj][m][n], 0, 0, 0); } while (0)
#define MMA(ai, bj, At, Bt, MID) do { __builtin_amdgcn_s_setprio(1); MMA_K(ai, bj, At, Bt, 0); SCHED; MID; SCHED; MMA_K(ai, bj, At, Bt, 1); __builtin_amdgcn_s_setprio(0); } while (0)
#define WAIT_V(n) asm volatile("s_waitcnt vmcnt(" #n ")" ::: "memory")
#define WAIT_VC(n2, n3) do { if (!comm) { if (helper) WAIT_V(n3); else WAIT_V(n2); } } while (0)
#define WAIT_L(n) asm volatile("s_waitcnt lgkmcnt(" #n ")" ::: "memory")
#define BAR __builtin_amdgcn_s_barrier()
#define SCHED __builtin_amdgcn_sched_barrier(0)

#define STAMP() do {} while (0)
#define STAMP2(i) do {} while (0)
    STAMP();
    f32x4 acc[2][2][4][2];
#pragma unroll
    for (int a = 0; a < 2; ++a)
#pragma unroll
        for (int b = 0; b < 2; ++b)
#pragma unroll
            for (int m = 0; m < 4; ++m)
#pragma unroll
                for (int n = 0; n < 2; ++n) acc[a][b][m][n] = (f32x4){0.f, 0.f, 0.f, 0.f};
    bf16x8 At[4][2], B0[2][2], B1[2][2];
    const char* cB = (const char*)P.w + (size_t)pn * 2 * hstepB;

    const int pi = (int)blockIdx.x >> 1, ph = (int)blockIdx.x & 1;
    const int npeer = ph ? 3 : 4;
    const unsigned xo0 = (unsigned)((4 * pi + (lane >> 4)) * K * 2 + (lane & 15) * 16);
    unsigned xo = xo0;
    const char* xsrc = (const char*)P.x + xo0;
    unsigned* const ctr = (unsigned*)(myws + CTL_OFF);
    unsigned* const tmo = (unsigned*)(myws + CTL_OFF + 256);
#define XLOAD(c) __builtin_amdgcn_global_load_lds((const unsigned*)(xsrc + (size_t)(c) * 256), (LAS unsigned*)(lds + XBUF + ((c) & 3) * 1024), 16, 0, 0)

    unsigned cv;
    asm volatile("s_nop 4\n\tglobal_load_dword %0, %1, %2 sc1" : "=v"(cv) : "v"(0u), "s"(ctr) : "memory");
    if (comm) { XLOAD(0); XLOAD(1); XLOAD(2); }
    WAIT_V(0);
    asm volatile("" : "+v"(cv));
    STAMP();
    const unsigned call = (unsigned)__builtin_amdgcn_readfirstlane(cv) / NWG + 1u;
    const unsigned fbase = call * 64u;
    const size_t par_off = (size_t)(call & 1u) * GATHER_BYTES;
    const size_t slot_off = par_off + (size_t)rank * MSH * K * 2;
    const int rot = ph ? pi % 3 : pi & 3;
    const char* pd[4];
#pragma unroll
    for (int k = 0; k < 4; ++k) { int q = k + rot; if (q >= npeer) q -= npeer; if (q >= npeer) q = 0;
        pd[k] = (const char*)P.ws[(rank + 1 + ph * 4 + q) & (NR - 1)] + slot_off; }
    const char* pflag;
    { const char* b = (const char*)P.ws[(rank + 1 + ph * 4) & (NR - 1)];
#pragma unroll
      for (int i = 1; i < 4; ++i) b = (lane == i && i < npeer) ? (const char*)P.ws[(rank + 1 + ph * 4 + i) & (NR - 1)] : b;
      pflag = b + FLAG_OFF + (size_t)rank * 512 + (size_t)pi * 4; }
    const unsigned long long fmask = ph ? 7ull : 15ull;
    const char* qflag = (const char*)myws + FLAG_OFF + (size_t)src * 512 + (size_t)(pm & 1) * 256 + (size_t)lane * 4;
    const unsigned lds0 = (unsigned)(size_t)lds;
    u32x4 xv = (u32x4){0u, 0u, 0u, 0u};
    unsigned pf = 0, tnow = 0;

    unsigned e_iss = 0, e_i0 = 0, e_i1 = 0, e_ts = 0;
#define AGG_TAU 240u
    int e_np = 0, e_r = 0, e_ack = 0, e_have = (remote && (comm || poller)) ? 0 : 1000;
    int e_act = 0, timed_out = 0;
#define VMCNT_NOW(dst) do { unsigned _v; asm volatile("s_getreg_b32 %0, hwreg(HW_REG_IB_STS, 0, 24)" : "=s"(_v) :: "memory"); dst = (_v & 15u) | ((_v >> 18) & 48u); } while (0)
#define DECIDE(lim, now) do { unsigned _c; VMCNT_NOW(_c); const unsigned _done = e_iss - _c; \
        if (e_np > 0 && (int)(_done - e_i0) >= 0) { STAMP2(e_ack); ++e_ack; --e_np; e_i0 = e_i1; const unsigned _fv = fbase + (unsigned)e_ack; \
            asm volatile("s_mov_b64 exec, %2\n\tglobal_store_dword %0, %1, off sc0 sc1\n\ts_mov_b64 exec, -1" :: "v"(pflag), "v"(_fv), "s"(fmask) : "memory"); ++e_iss; } \
        { unsigned _dt = (now) - e_ts; _dt = _dt < (1u << 20) ? _dt : (1u << 20); const int _tl = (int)((_dt * (65536u / AGG_TAU)) >> 16); \
          const int _m = (_tl + 1 < (lim)) ? _tl + 1 : (lim); e_act = (int)((((unsigned)(e_r - _m)) & ((unsigned)(e_np - 2))) >> 31); } \
    } while (0)
#define NOW() ((unsigned)__builtin_amdgcn_s_memrealtime())
#define POLL() __builtin_amdgcn_global_load_lds((const unsigned*)qflag, (LAS unsigned*)(lds + PBUF), 4, 0, 17)
#define PREAD() asm volatile("ds_read_b32 %0, %1" : "=v"(pf) : "v"(lds0 + PBUF + (unsigned)lane * 4u) : "memory")
#define PEVAL() do { asm volatile("" : "+v"(pf)); const unsigned _n1 = fbase + (unsigned)e_have + 1u; \
        if (__all(pf >= _n1)) { STAMP2(32 + e_have); ++e_have; if (__all(pf > _n1)) { STAMP2(32 + e_have); ++e_have; } } } while (0)
#define EVAL() do { PREAD(); WAIT_L(0); PEVAL(); } while (0)
#define XREAD(c) asm volatile("ds_read_b128 %0, %1" : "=v"(xv) : "v"(lds0 + XBUF + (unsigned)((c) & 3) * 1024u + (unsigned)lane * 16u) : "memory")
#define PST(k) do { asm volatile("s_nop 4\n\tglobal_store_dwordx4 %0, %1, %2 sc0 sc1\n\ts_nop 1" :: "v"(xo), "v"(xv), "s"(pd[k]) : "memory"); ++e_iss; } while (0)
#define SLOT(k) do { asm volatile("" : "+s"(e_act)); if (e_act != 0) { \
        if ((k) == 0 && e_r + 3 < NCH) { XLOAD(e_r + 3); ++e_iss; } \
        if ((k) < 3 || npeer == 4) PST(k); \
        if ((k) == 3) { if (e_np == 0) e_i0 = e_iss; e_i1 = e_iss; ++e_np; ++e_r; xo += 256u; e_act = 0; XREAD(e_r); } } } while (0)
#define TICK(lim) do { DECIDE(lim, NOW()); WAIT_L(0); SLOT(0); SLOT(1); SLOT(2); SLOT(3); } while (0)
#define BOUNDED_UNTIL(cond, BODY) do { if (!(cond)) { const unsigned long long _t0 = __builtin_amdgcn_s_memrealtime(); unsigned _sp = 0; \
        while (!(cond)) { BODY; \
            if ((++_sp & 63u) == 0u) { if (__builtin_amdgcn_s_memrealtime() - _t0 > DEADLINE_TICKS) { timed_out = 1; break; } } \
            __builtin_amdgcn_s_sleep(1); } \
        if (timed_out != 0) { if (lane == 0) asm volatile("s_nop 4\n\tglobal_store_dword %0, %1, %2 sc1\n\ts_waitcnt vmcnt(0)" :: "v"(0u), "v"(call), "s"(tmo) : "memory"); ++e_iss; e_have = 1000; } } } while (0)
#define GATE(need, lim) do { if (e_have < (need)) { \
        if (comm) { SLOT(2); SLOT(3); BOUNDED_UNTIL(e_have >= (need), { EVAL(); TICK(lim); }); } \
        else { BOUNDED_UNTIL(e_have >= (need), { POLL(); WAIT_V(0); EVAL(); }); } } } while (0)

    if (poller) { asm volatile("ds_write_b32 %0, %1\n\ts_waitcnt lgkmcnt(0)" :: "v"(lds0 + PBUF + (unsigned)lane * 4u), "v"(0u) : "memory"); }
    e_ts = (unsigned)__builtin_amdgcn_s_memrealtime();
    if (comm) { XREAD(0); TICK(DLA); }
    BAR;
    STAGE_B(SB(0, 0), cB); HELP_B(SB(0, 0), cB); STAGE_B(SB(0, 1), cB + hstepB); HELP_B(SB(0, 1), cB + hstepB);
    STAGE_B(SB(1, 0), cB + kstepB); HELP_B(SB(1, 0), cB + kstepB); STAGE_B(SB(1, 1), cB + hstepB + kstepB); HELP_B(SB(1, 1), cB + hstepB + kstepB);
    GATE(1, DLA);
#define AGG_LW 2
    if (comm && !remote) BOUNDED_UNTIL(e_ack >= AGG_LW, TICK(DLA));
    STAMP();
    const char* cA = remote ? (const char*)myws + par_off + (size_t)pm * 2 * hstepA : (const char*)P.x + (size_t)(pm & 1) * 2 * hstepA;
    BAR;
    STAGE_A(SA(0, 0), cA); HELP_A(SA(0, 0), cA); STAGE_A(SA(0, 1), cA + hstepA); HELP_A(SA(0, 1), cA + hstepA); STAGE_A(SA(1, 0), cA + kstepA); HELP_A(SA(1, 0), cA + kstepA);
    if (comm) TICK(DLA);
    if (poller && remote) { POLL(); WAIT_V(3); } else WAIT_VC(2, 3);
    BAR;
    if (comm) DECIDE(DLA, NOW());
    if (wr == 1) BAR;
    STAMP();
    for (int t = 0; t < nt; t += 2) {
        const int it = t >> 1;
        const bool last = (t == nt - 2);
        const char* a1 = cA + (size_t)(t + 1) * kstepA;
        const int t2 = last ? 0 : t + 2;
        const char* a2 = cA + (size_t)t2 * kstepA; const char* b2 = cB + (size_t)t2 * kstepB;
        const char* a3 = a2 + kstepA; const char* b3 = b2 + kstepB;
        const int want = (it + 4 < NCH) ? it + 4 : NCH;
        const int need = (it + 2 < NCH) ? it + 2 : NCH;
        const int lim = (it + DLA < NCH) ? it + DLA : NCH;
        const int lim8 = (it + 1 + DLA < NCH) ? it + 1 + DLA : NCH;
        if (comm) WAIT_L(0);
        if (e_have < want) PREAD();
        LDB(B0, 0, 0); SCHED; LDA(At, 0, 0); STAGE_A(SA(1, 1), a1 + hstepA);
        SLOT(0);
        WAIT_L(8); BAR; WAIT_L(0); SCHED; MMA(0, 0, At, B0, HELP_A(SA(1, 1), a1 + hstepA)); BAR; SCHED;
        LDB(B1, 0, 1); STAGE_B(SB(0, 0), b2);
        SLOT(1);
        if (e_have < want) PEVAL();
        GATE(need, lim);
        BAR; WAIT_L(0); SCHED; MMA(0, 1, At, B1, HELP_B(SB(0, 0), b2)); BAR;
        LDA(At, 0, 1); STAGE_A(SA(0, 0), a2);
        SLOT(2); if (comm) tnow = NOW();
        BAR; WAIT_L(0); SCHED; MMA(1, 0, At, B0, HELP_A(SA(0, 0), a2)); BAR; SCHED;
        STAGE_B(SB(0, 1), b2 + hstepB);
        SLOT(3); if (comm) DECIDE(lim, tnow);
        WAIT_VC(6, 8);
        BAR; MMA(1, 1, At, B1, HELP_B(SB(0, 1), b2 + hstepB)); BAR;
        if (comm) WAIT_L(0);
        LDB(B0, 1, 0); SCHED; LDA(At, 1, 0); STAGE_A(SA(0, 1), a2 + hstepA);
        SLOT(0); if (comm) { if (e_have < want) { POLL(); ++e_iss; } }
        WAIT_L(8); BAR; WAIT_L(0); SCHED; MMA(0, 0, At, B0, HELP_A(SA(0, 1), a2 + hstepA)); BAR; SCHED;
        LDB(B1, 1, 1); STAGE_B(SB(1, 0), b3);
        SLOT(1);
        BAR; WAIT_L(0); SCHED; MMA(0, 1, At, B1, HELP_B(SB(1, 0), b3)); BAR;
        LDA(At, 1, 1); STAGE_A(SA(1, 0), a3);
        SLOT(2); if (comm) tnow = NOW();
        BAR; WAIT_L(0); SCHED; MMA(1, 0, At, B0, HELP_A(SA(1, 0), a3)); BAR; SCHED;
        STAGE_B(SB(1, 1), b3 + hstepB);
        SLOT(3); if (comm) DECIDE(lim8, tnow);
        WAIT_VC(6, 8);
        BAR; MMA(1, 1, At, B1, HELP_B(SB(1, 1), b3 + hstepB)); BAR;
        STAMP();
    }
    if (wr == 0) BAR;
    if (comm) { WAIT_L(0); SLOT(0); SLOT(1); SLOT(2); SLOT(3); BOUNDED_UNTIL(e_ack >= NCH, TICK(NCH)); }
    {
        unsigned tv; asm volatile("s_nop 4\n\tglobal_load_dword %0, %1, %2 sc1\n\ts_waitcnt vmcnt(0)" : "=v"(tv) : "v"(0u), "s"(tmo) : "memory");
        const bool bad = ((unsigned)__builtin_amdgcn_readfirstlane(tv) == call);
        const int row0 = pm * BM + wr * 64 + fr, col0 = pn * BM + wc * 32 + (fq & 1) * 16 + (fq >> 1) * 8;
#pragma unroll
        for (int ai = 0; ai < 2; ++ai)
#pragma unroll
            for (int m = 0; m < 4; ++m) { bf16_t* rowp = P.out + (size_t)(row0 + ai * HALF + m * 16) * N + col0;
#pragma unroll
                for (int bj = 0; bj < 2; ++bj) { const f32x4 v0 = acc[ai][bj][m][0], v1 = acc[ai][bj][m][1];
                    const unsigned ax = cvt_pk_bf16(silu_f(v0[0]), silu_f(v0[1])), ay = cvt_pk_bf16(silu_f(v0[2]), silu_f(v0[3]));
                    const unsigned bx = cvt_pk_bf16(silu_f(v1[0]), silu_f(v1[1])), by = cvt_pk_bf16(silu_f(v1[2]), silu_f(v1[3]));
                    const u32x2 sx = __builtin_amdgcn_permlane16_swap(ax, bx, false, false), sy = __builtin_amdgcn_permlane16_swap(ay, by, false, false);
                    u32x4 w; w.x = sx.x; w.y = sy.x; w.z = sx.y; w.w = sy.y;
                    if (bad) { w.x = 0x7fc07fc0u; w.y = 0x7fc07fc0u; w.z = 0x7fc07fc0u; w.w = 0x7fc07fc0u; }
                    *(u32x4*)(rowp + bj * HALF) = w; } }
    }
    if (tid == 0) __hip_atomic_fetch_add(ctr, 1u, __ATOMIC_RELAXED, __HIP_MEMORY_SCOPE_AGENT);
    WAIT_V(0);
}
}

extern "C" void kernel_launch(int rank, int nranks, ncclComm_t comm,
                              hipStream_t stream,
                              void* const* d_in, const int* in_sizes, int n_in,
                              void* d_out, int out_size,
                              const int* in_dtypes, int out_dtype,
                              void* const* d_out_all,
                              void* const* d_ws_all, size_t ws_bytes) {
    using namespace agg;
    static bool init = false;
    if (!init) { (void)hipFuncSetAttribute((const void*)ag_gemm_k, hipFuncAttributeMaxDynamicSharedMemorySize, STAGE_BYTES); init = true; }
    if (nranks != NR || ws_bytes < WS_NEED) { fprintf(stderr, "unsupported configuration\n"); return; }
    Params p;
    p.x = (const bf16_t*)d_in[0]; p.w = (const bf16_t*)d_in[1]; p.out = (bf16_t*)d_out; p.rank = rank;
    for (int i = 0; i < NR; ++i) p.ws[i] = (unsigned char*)d_ws_all[i];
    ag_gemm_k<<<NWG, 512, STAGE_BYTES, stream>>>(p);
}
```

```cpp
#include <hip/hip_runtime.h>
#include <rccl/rccl.h>
#include <cstdio>

namespace agg {
#define LAS __attribute__((address_space(3)))
typedef unsigned short bf16_t;
typedef short bf16x8 __attribute__((ext_vector_type(8)));
typedef short s16x4 __attribute__((ext_vector_type(4)));
typedef float f32x4 __attribute__((ext_vector_type(4)));
typedef unsigned u32x4 __attribute__((ext_vector_type(4)));
typedef unsigned u32x2 __attribute__((ext_vector_type(2)));

constexpr int NR = 8;
constexpr int MM = 4096, NN = 4096, KK = 4096, MSH = MM / NR;
constexpr int BM = 256, BK = 64, HALF = 128, HTB = HALF * BK * 2, GEMM_LDS = 8 * HTB, NXCD = 8, WGM = 8;
constexpr int XBUF = GEMM_LDS, PBUF = GEMM_LDS + 4096, STAGE_BYTES = GEMM_LDS + 4096 + 512;
constexpr int NWG = 256;
constexpr int NCH = 32;
#define AGG_D 8
constexpr int DLA = AGG_D;
constexpr size_t GATHER_BYTES = (size_t)MM * KK * 2;
constexpr size_t FLAG_OFF = 2 * GATHER_BYTES;
constexpr size_t CTL_OFF = FLAG_OFF + 64 * 1024;
constexpr size_t WS_NEED = CTL_OFF + 4096;
constexpr unsigned long long DEADLINE_TICKS = 50000000ull;

template <int V> struct IC { static constexpr int value = V; };
struct Params { const bf16_t* x; const bf16_t* w; bf16_t* out; unsigned char* ws[NR]; int rank; };

__host__ __device__ __forceinline__ int lds_byte(int r, int c) { const int st = (r >> 4) * 2 + (c >> 5), rr = r & 15, cc = c & 31, ob = rr * 64 + cc * 2; return st * 1024 + (ob ^ (((ob >> 9) & 1) << 5)); }
__host__ __device__ __forceinline__ void stage_rc(int b, int& R, int& C) { const int st = b / 1024, sb = b % 1024, swz = sb ^ (((sb >> 9) & 1) << 5); R = (st >> 1) * 16 + swz / 64; C = (st & 1) * 32 + (swz % 64) / 2; }
__host__ __device__ __forceinline__ int bswz(int row) { return ((row & 3) | (((row >> 3) & 1) << 2)) << 1; }

__device__ __forceinline__ unsigned cvt_pk_bf16(float lo, float hi) { unsigned r; asm volatile("v_cvt_pk_bf16_f32 %0, %1, %2" : "=v"(r) : "v"(lo), "v"(hi)); return r; }
__device__ __forceinline__ float silu_f(float y) {
    const float c = fminf(fmaxf(y, -60.f), 60.f);
    const float e = __builtin_amdgcn_exp2f(c * -1.44269504088896340736f);
    return y * __builtin_amdgcn_rcpf(1.0f + e);
}

__global__ __launch_bounds__(512, 2) void ag_gemm_k(const Params P) {
    extern __shared__ __attribute__((aligned(16))) unsigned char shm[];
    LAS unsigned char* lds = (LAS unsigned char*)shm;
    const int tid = threadIdx.x, wid = __builtin_amdgcn_readfirstlane(tid >> 6), lane = tid & 63, wr = wid >> 2, wc = wid & 3, fr = lane & 15, fq = lane >> 4;
    constexpr int K = KK, N = NN, nt = K / BK, nN = NN / BM;
    int wgid = blockIdx.x; { const int q = NWG / NXCD, xcd = wgid % NXCD, off = wgid / NXCD; wgid = xcd * q + off; }
    const int nig = WGM * nN, gid = wgid / nig, fm = gid * WGM;
    const int pm = fm + ((wgid % nig) % WGM), pn = (wgid % nig) / WGM;
    const int rank = P.rank, src = pm >> 1;
    const bool remote = (src != rank);
    unsigned char* const myws = P.ws[rank];
    const bool comm = (wid == 7);
#define AGG_H2 2
#define AGG_W1 2
    const bool helper = (wid == 3 || wid == AGG_H2);
    const bool poller = (wid == 4);

#define SA(b, h) (((b) * 2 + (h)) * HTB)
#define SB(b, h) ((4 + (b) * 2 + (h)) * HTB)
    unsigned voffA[2], voffB;
#pragma unroll
    for (int i = 0; i < 2; ++i) { int R, C; stage_rc(tid * 16 + i * 8192, R, C); voffA[i] = (unsigned)(R * K + C) * 2u; }
    { const int b = tid * 16, row = b >> 8, chp = (b >> 4) & 15, ch = chp ^ bswz(row); voffB = (unsigned)(row * N * 2 + ch * 16); }
    const size_t kstepA = (size_t)(BK * 2), kstepB = (size_t)BK * N * 2;
    const size_t hstepA = (size_t)HALF * K * 2, hstepB = (size_t)HALF * 2;
    const unsigned ldsw = (unsigned)wid * 1024u;
    const unsigned hA = (wid == 3) ? 32u * K * 2u : (7u - (AGG_H2 >> 1)) * 16u * K * 2u + ((AGG_H2 & 1) ? 0u : 64u);
    const unsigned hB = (wid == 3) ? 16u * N * 2u : (15u - AGG_H2) * 4u * N * 2u;
    const unsigned hL = (wid == 3) ? 7u * 1024u : 15u * 1024u;
    const int aoff = lds_byte(wr * 64 + fr, fq * 8);
    int boff[2];
    { const int g = fq, q = (lane & 15) >> 2, p = lane & 3, row0 = 8 * g + q;
#pragma unroll
      for (int n = 0; n < 2; ++n) { const int c = 2 * wc + n; boff[n] = (int)(unsigned)(size_t)lds + SB(0, 0) + 256 * row0 + 16 * ((2 * c + (p >> 1)) ^ bswz(row0)) + 8 * (p & 1); } }
#define GLDS(g, l) __builtin_amdgcn_global_load_lds((const unsigned*)(g), (LAS unsigned*)(l), 16, 0, 0)
#define STAGE_A(bufoff, gbase) do { if (!comm) { _Pragma("unroll") for (int _i = 0; _i < 2; ++_i) GLDS((const char*)(gbase) + voffA[_i], lds + (bufoff) + ldsw + _i * 8192); } } while (0)
#define STAGE_B(bufoff, gbase) do { if (!comm) { _Pragma("unroll") for (int _i = 0; _i < 2; ++_i) GLDS((const char*)(gbase) + (size_t)_i * 32 * N * 2 + voffB, lds + (bufoff) + ldsw + _i * 8192); } } while (0)
#define HELP_A(bufoff, gbase) do { if (helper) GLDS((const char*)(gbase) + hA + voffA[0], lds + (bufoff) + hL); } while (0)
#define HELP_B(bufoff, gbase) do { if (helper) GLDS((const char*)(gbase) + hB + voffB, lds + (bufoff) + hL); } while (0)
#define LDA(dst, b, h) do { _Pragma("unroll") for (int m = 0; m < 4; ++m) _Pragma("unroll") for (int k = 0; k < 2; ++k) dst[m][k] = *(const LAS bf16x8*)(lds + SA(b, h) + aoff + m * 2048 + k * 1024); } while (0)
#define TRRD(dst, addr, off) asm volatile("ds_read_b64_tr_b16 %0, %1 offset:%2" : "=v"(dst) : "v"(addr), "i"(off) : "memory")
#define LDB(dst, b, h) do { _Pragma("unroll") for (int n = 0; n < 2; ++n) _Pragma("unroll") for (int k = 0; k < 2; ++k) { \
        s16x4 lo, hi; TRRD(lo, boff[n], ((b) * 2 + (h)) * HTB + k * 8192); TRRD(hi, boff[n], ((b) * 2 + (h)) * HTB + k * 8192 + 1024); \
        dst[n][k] = __builtin_shufflevector(lo, hi, 0, 1, 2, 3, 4, 5, 6, 7); } } while (0)
#define MMA_K(ai, bj, At, Bt, k) do { _Pragma("unroll") for (int m = 0; m < 4; ++m) _Pragma("unroll") for (int n = 0; n < 2; ++n) \
        acc[ai][bj][m][n] = __builtin_amdgcn_mfma_f32_16x16x32_bf16(Bt[n][k], At[m][k], acc[ai][bj][m][n], 0, 0, 0); } while (0)
#define MMA(ai, bj, At, Bt, MID) do { __builtin_amdgcn_s_setprio(1); MMA_K(ai, bj, At, Bt, 0); SCHED; MID; SCHED; MMA_K(ai, bj, At, Bt, 1); __builtin_amdgcn_s_setprio(0); } while (0)
#define WAIT_V(n) asm volatile("s_waitcnt vmcnt(" #n ")" ::: "memory")
#define WAIT_VC(n2, n3) do { if (!comm) { if (helper) WAIT_V(n3); else WAIT_V(n2); } } while (0)
#define WAIT_L(n) asm volatile("s_waitcnt lgkmcnt(" #n ")" ::: "memory")
#define BAR __builtin_amdgcn_s_barrier()
#define SCHED __builtin_amdgcn_sched_barrier(0)

#define STAMP() do {} while (0)
#define STAMP2(i) do {} while (0)
    STAMP();
    f32x4 acc[2][2][4][2];
#pragma unroll
    for (int a = 0; a < 2; ++a)
#pragma unroll
        for (int b = 0; b < 2; ++b)
#pragma unroll
            for (int m = 0; m < 4; ++m)
#pragma unroll
                for (int n = 0; n < 2; ++n) acc[a][b][m][n] = (f32x4){0.f, 0.f, 0.f, 0.f};
    bf16x8 At[4][2], B0[2][2], B1[2][2];
    const char* cB = (const char*)P.w + (size_t)pn * 2 * hstepB;

    const int pi = (int)blockIdx.x >> 1, ph = (int)blockIdx.x & 1;
    const int npeer = ph ? 3 : 4;
    const unsigned xo0 = (unsigned)((4 * pi + (lane >> 4)) * K * 2 + (lane & 15) * 16);
    unsigned xo = xo0;
    const char* xsrc = (const char*)P.x + xo0;
    unsigned* const ctr = (unsigned*)(myws + CTL_OFF);
    unsigned* const tmo = (unsigned*)(myws + CTL_OFF + 256);
#define XLOAD(c) __builtin_amdgcn_global_load_lds((const unsigned*)(xsrc + (size_t)(c) * 256), (LAS unsigned*)(lds + XBUF + ((c) & 3) * 1024), 16, 0, 0)

    unsigned cv;
    asm volatile("s_nop 4\n\tglobal_load_dword %0, %1, %2 sc1" : "=v"(cv) : "v"(0u), "s"(ctr) : "memory");
    if (comm) { XLOAD(0); XLOAD(1); XLOAD(2); }
    WAIT_V(0);
    asm volatile("" : "+v"(cv));
    STAMP();
    const unsigned call = (unsigned)__builtin_amdgcn_readfirstlane(cv) / NWG + 1u;
    const unsigned fbase = call * 64u;
    const size_t par_off = (size_t)(call & 1u) * GATHER_BYTES;
    const size_t slot_off = par_off + (size_t)rank * MSH * K * 2;
    const int rot = ph ? pi % 3 : pi & 3;
    const char* pd[4];
#pragma unroll
    for (int k = 0; k < 4; ++k) { int q = k + rot; if (q >= npeer) q -= npeer; if (q >= npeer) q = 0;
        pd[k] = (const char*)P.ws[(rank + 1 + ph * 4 + q) & (NR - 1)] + slot_off; }
    const char* pflag;
    { const char* b = (const char*)P.ws[(rank + 1 + ph * 4) & (NR - 1)];
#pragma unroll
      for (int i = 1; i < 4; ++i) b = (lane == i && i < npeer) ? (const char*)P.ws[(rank + 1 + ph * 4 + i) & (NR - 1)] : b;
      pflag = b + FLAG_OFF + (size_t)rank * 512 + (size_t)pi * 4; }
    const unsigned long long fmask = ph ? 7ull : 15ull;
    const char* qflag = (const char*)myws + FLAG_OFF + (size_t)src * 512 + (size_t)(pm & 1) * 256 + (size_t)lane * 4;
    const unsigned lds0 = (unsigned)(size_t)lds;
    u32x4 xv = (u32x4){0u, 0u, 0u, 0u};
    unsigned pf = 0, tnow = 0;

    unsigned e_iss = 0, e_i0 = 0, e_i1 = 0, e_due = 0;
#define AGG_TAU 240u
    int e_np = 0, e_r = 0, e_ack = 0, e_have = (remote && (comm || poller)) ? 0 : 1000;
    int e_act = 0, timed_out = 0;
#define VMCNT_NOW(dst) do { unsigned _v; asm volatile("s_getreg_b32 %0, hwreg(HW_REG_IB_STS, 0, 24)" : "=s"(_v) :: "memory"); dst = (_v & 15u) | ((_v >> 18) & 48u); } while (0)
#define DECIDE(lim, now) do { unsigned _c; VMCNT_NOW(_c); const unsigned _done = e_iss - _c; \
        if (e_np > 0 && (int)(_done - e_i0) >= 0) { STAMP2(e_ack); ++e_ack; --e_np; e_i0 = e_i1; const unsigned _fv = fbase + (unsigned)e_ack; \
            asm volatile("s_mov_b64 exec, %2\n\tglobal_store_dword %0, %1, off sc0 sc1\n\ts_mov_b64 exec, -1" :: "v"(pflag), "v"(_fv), "s"(fmask) : "memory"); ++e_iss; } \
        e_act = (int)((((unsigned)(e_r - (lim))) & ((unsigned)(e_np - 2)) & ~((now) - e_due)) >> 31); \
    } while (0)
#define NOW() ((unsigned)__builtin_amdgcn_s_memrealtime())
#define PSLEEP do {} while (0)
#define POLL() __builtin_amdgcn_global_load_lds((const unsigned*)qflag, (LAS unsigned*)(lds + PBUF), 4, 0, 17)
#define PREAD() asm volatile("ds_read_b32 %0, %1" : "=v"(pf) : "v"(lds0 + PBUF + (unsigned)lane * 4u) : "memory")
#define PEVAL() do { asm volatile("" : "+v"(pf)); const unsigned _n1 = fbase + (unsigned)e_have + 1u; \
        if (__all(pf >= _n1)) { STAMP2(32 + e_have); ++e_have; if (__all(pf > _n1)) { STAMP2(32 + e_have); ++e_have; } } } while (0)
#define EVAL() do { PREAD(); WAIT_L(0); PEVAL(); } while (0)
#define XREAD(c) asm volatile("ds_read_b128 %0, %1" : "=v"(xv) : "v"(lds0 + XBUF + (unsigned)((c) & 3) * 1024u + (unsigned)lane * 16u) : "memory")
#define PST(k) do { asm volatile("s_nop 4\n\tglobal_store_dwordx4 %0, %1, %2 sc0 sc1\n\ts_nop 1" :: "v"(xo), "v"(xv), "s"(pd[k]) : "memory"); ++e_iss; } while (0)
#define SLOT(k) do { asm volatile("" : "+s"(e_act)); if (e_act != 0) { \
        if ((k) == 0 && e_r + 3 < NCH) { XLOAD(e_r + 3); ++e_iss; } \
        if ((k) < 3 || npeer == 4) PST(k); \
        if ((k) == 3) { if (e_np == 0) e_i0 = e_iss; e_i1 = e_iss; ++e_np; ++e_r; e_due += AGG_TAU; xo += 256u; e_act = 0; XREAD(e_r); } } } while (0)
#define TICK(lim) do { DECIDE(lim, NOW()); WAIT_L(0); SLOT(0); SLOT(1); SLOT(2); SLOT(3); } while (0)
#define BOUNDED_UNTIL(cond, BODY) do { if (!(cond)) { const unsigned long long _t0 = __builtin_amdgcn_s_memrealtime(); unsigned _sp = 0; \
        while (!(cond)) { BODY; \
            if ((++_sp & 63u) == 0u) { if (__builtin_amdgcn_s_memrealtime() - _t0 > DEADLINE_TICKS) { timed_out = 1; break; } } \
            __builtin_amdgcn_s_sleep(1); } \
        if (timed_out != 0) { if (lane == 0) asm volatile("s_nop 4\n\tglobal_store_dword %0, %1, %2 sc1\n\ts_waitcnt vmcnt(0)" :: "v"(0u), "v"(call), "s"(tmo) : "memory"); ++e_iss; e_have = 1000; } } } while (0)
#define GATE(need, lim) do { if (e_have < (need)) { \
        if (GROLE == 3) { SLOT(2); SLOT(3); BOUNDED_UNTIL(e_have >= (need), { EVAL(); TICK(lim); }); } \
        else { BOUNDED_UNTIL(e_have >= (need), { POLL(); WAIT_V(0); EVAL(); PSLEEP; }); } } } while (0)

    if (poller) { asm volatile("ds_write_b32 %0, %1\n\ts_waitcnt lgkmcnt(0)" :: "v"(lds0 + PBUF + (unsigned)lane * 4u), "v"(0u) : "memory"); }
#define AGG_PHI 256
    e_due = (unsigned)__builtin_amdgcn_s_memrealtime() + (unsigned)blockIdx.x * AGG_TAU * AGG_PHI / 65536u;
    if (comm) { XREAD(0); TICK(DLA); }
    BAR;
    STAGE_B(SB(0, 0), cB); HELP_B(SB(0, 0), cB); STAGE_B(SB(0, 1), cB + hstepB); HELP_B(SB(0, 1), cB + hstepB);
    STAGE_B(SB(1, 0), cB + kstepB); HELP_B(SB(1, 0), cB + kstepB); STAGE_B(SB(1, 1), cB + hstepB + kstepB); HELP_B(SB(1, 1), cB + hstepB + kstepB);
#define GROLE (comm ? 3 : 2)
    GATE(1, DLA);
#undef GROLE
#define GROLE ROLE
#define AGG_LW 2
    if (comm && !remote) BOUNDED_UNTIL(e_ack >= AGG_LW, TICK(DLA));
    STAMP();
    const char* cA = remote ? (const char*)myws + par_off + (size_t)pm * 2 * hstepA : (const char*)P.x + (size_t)(pm & 1) * 2 * hstepA;
    BAR;
    STAGE_A(SA(0, 0), cA); HELP_A(SA(0, 0), cA); STAGE_A(SA(0, 1), cA + hstepA); HELP_A(SA(0, 1), cA + hstepA); STAGE_A(SA(1, 0), cA + kstepA); HELP_A(SA(1, 0), cA + kstepA);
    if (comm) TICK(DLA);
    if (poller && remote) { POLL(); WAIT_V(3); } else WAIT_VC(2, 3);
    BAR;
    if (comm) { tnow = NOW(); }
    if (wr == 1) BAR;
    STAMP();
#define RSTAGE_A(bufoff, gbase) do { if (ROLE != 3) { _Pragma("unroll") for (int _i = 0; _i < 2; ++_i) GLDS((const char*)(gbase) + voffA[_i], lds + (bufoff) + ldsw + _i * 8192); } } while (0)
#define RSTAGE_B(bufoff, gbase) do { if (ROLE != 3) { _Pragma("unroll") for (int _i = 0; _i < 2; ++_i) GLDS((const char*)(gbase) + (size_t)_i * 32 * N * 2 + voffB, lds + (bufoff) + ldsw + _i * 8192); } } while (0)
#define RHELP_A(bufoff, gbase) do { if (ROLE == 1) GLDS((const char*)(gbase) + hA + voffA[0], lds + (bufoff) + hL); } while (0)
#define RHELP_B(bufoff, gbase) do { if (ROLE == 1) GLDS((const char*)(gbase) + hB + voffB, lds + (bufoff) + hL); } while (0)
#define RWAIT() do { if (ROLE == 1) WAIT_V(8); else if (ROLE != 3) WAIT_V(6); } while (0)
#define RSLOT(k) do { if (ROLE == 3) SLOT(k); } while (0)
    auto kloop = [&](auto role_c, const int t_beg, const int t_end) __attribute__((always_inline)) {
      constexpr bool EAGER = decltype(role_c)::value == 4;
      constexpr int ROLE = EAGER ? 3 : decltype(role_c)::value;
      for (int t = t_beg; t < t_end; t += 2) {
        const int it = t >> 1;
        const bool last = (t == nt - 2);
        const char* a1 = cA + (size_t)(t + 1) * kstepA;
        const int t2 = last ? 0 : t + 2;
        const char* a2 = cA + (size_t)t2 * kstepA; const char* b2 = cB + (size_t)t2 * kstepB;
        const char* a3 = a2 + kstepA; const char* b3 = b2 + kstepB;
        const int want = (it + 4 < NCH) ? it + 4 : NCH;
        const int need = (it + 2 < NCH) ? it + 2 : NCH;
        const int lim = (it + DLA < NCH) ? it + DLA : NCH;
        const int lim8 = (it + 1 + DLA < NCH) ? it + 1 + DLA : NCH;
#define AGG_VA 0
#define AGG_VB 1
#define ASLOT(k) do { if (ROLE == 3 && !EAGER) SLOT(k); } while (0)
#define ETICK() do { if (EAGER) { WAIT_L(0); unsigned _c2; VMCNT_NOW(_c2); \
            const int _y = e_np > 0 ? (int)(e_iss - _c2 - e_i0) : -1; \
            const int _x = (int)((tnow - e_due) | ~(((unsigned)(e_r - lim)) & ((unsigned)(e_np - 2)))); \
            if ((_x & _y) >= 0) { DECIDE(lim, tnow); SLOT(0); SLOT(1); SLOT(2); SLOT(3); } \
            tnow = NOW(); } } while (0)
        ETICK();
        if (ROLE == 3) WAIT_L(0);
        if (ROLE >= 2 && !AGG_VB) PREAD();
        LDB(B0, 0, 0); SCHED; LDA(At, 0, 0); RSTAGE_A(SA(1, 1), a1 + hstepA);
        ASLOT(0);
        WAIT_L(8); BAR; WAIT_L(0); SCHED; MMA(0, 0, At, B0, RHELP_A(SA(1, 1), a1 + hstepA)); BAR; SCHED;
        ETICK();
        if (ROLE >= 2 && AGG_VB) PREAD();
        LDB(B1, 0, 1); RSTAGE_B(SB(0, 0), b2);
        ASLOT(1);
        if (ROLE >= 2) { if (AGG_VB) WAIT_L(8); if (e_have < want) PEVAL();
          GATE(need, lim);
        }
        BAR; WAIT_L(0); SCHED; MMA(0, 1, At, B1, RHELP_B(SB(0, 0), b2)); BAR;
        ETICK();
        LDA(At, 0, 1); RSTAGE_A(SA(0, 0), a2);
        ASLOT(2); if (ROLE == 3 && !EAGER) tnow = NOW();
        BAR; WAIT_L(0); SCHED; MMA(1, 0, At, B0, RHELP_A(SA(0, 0), a2)); BAR; SCHED;
        ETICK();
        RSTAGE_B(SB(0, 1), b2 + hstepB);
        ASLOT(3); if (ROLE == 3 && !EAGER) DECIDE(lim, tnow);
        RWAIT();
        BAR; MMA(1, 1, At, B1, RHELP_B(SB(0, 1), b2 + hstepB)); BAR;
        ETICK();
        if (ROLE == 3) WAIT_L(0);
        LDB(B0, 1, 0); SCHED; LDA(At, 1, 0); RSTAGE_A(SA(0, 1), a2 + hstepA);
        ASLOT(0); if (ROLE == 3 && !AGG_VB) { if (e_have < want) { POLL(); ++e_iss; } }
        WAIT_L(8); BAR; WAIT_L(0); SCHED; MMA(0, 0, At, B0, RHELP_A(SA(0, 1), a2 + hstepA)); BAR; SCHED;
        ETICK();
        LDB(B1, 1, 1); RSTAGE_B(SB(1, 0), b3);
        ASLOT(1);
        BAR; WAIT_L(0); SCHED; MMA(0, 1, At, B1, RHELP_B(SB(1, 0), b3)); BAR;
        ETICK();
        LDA(At, 1, 1); RSTAGE_A(SA(1, 0), a3);
        ASLOT(2); if (ROLE == 3) { if (AGG_VB) { if (e_have < want + 1) { POLL(); ++e_iss; } } if (!EAGER) tnow = NOW(); }
        BAR; WAIT_L(0); SCHED; MMA(1, 0, At, B0, RHELP_A(SA(1, 0), a3)); BAR; SCHED;
        ETICK();
        RSTAGE_B(SB(1, 1), b3 + hstepB);
        ASLOT(3); if (ROLE == 3 && !EAGER) DECIDE(lim8, tnow);
        RWAIT();
        BAR; MMA(1, 1, At, B1, RHELP_B(SB(1, 1), b3 + hstepB)); BAR;
        STAMP();
      }
    };
#define AGG_NE 4
    if (comm) { kloop(IC<4>{}, 0, 2 * AGG_NE); kloop(IC<3>{}, 2 * AGG_NE, nt); } else if (helper) kloop(IC<1>{}, 0, nt); else if (poller) kloop(IC<2>{}, 0, nt); else kloop(IC<0>{}, 0, nt);
    if (wr == 0) BAR;
    if (comm) { WAIT_L(0); SLOT(0); SLOT(1); SLOT(2); SLOT(3); BOUNDED_UNTIL(e_ack >= NCH, TICK(NCH)); }
    {
        unsigned tv; asm volatile("s_nop 4\n\tglobal_load_dword %0, %1, %2 sc1\n\ts_waitcnt vmcnt(0)" : "=v"(tv) : "v"(0u), "s"(tmo) : "memory");
        const bool bad = ((unsigned)__builtin_amdgcn_readfirstlane(tv) == call);
        const int row0 = pm * BM + wr * 64 + fr, col0 = pn * BM + wc * 32 + (fq & 1) * 16 + (fq >> 1) * 8;
#pragma unroll
        for (int ai = 0; ai < 2; ++ai)
#pragma unroll
            for (int m = 0; m < 4; ++m) { bf16_t* rowp = P.out + (size_t)(row0 + ai * HALF + m * 16) * N + col0;
#pragma unroll
                for (int bj = 0; bj < 2; ++bj) { const f32x4 v0 = acc[ai][bj][m][0], v1 = acc[ai][bj][m][1];
                    const unsigned ax = cvt_pk_bf16(silu_f(v0[0]), silu_f(v0[1])), ay = cvt_pk_bf16(silu_f(v0[2]), silu_f(v0[3]));
                    const unsigned bx = cvt_pk_bf16(silu_f(v1[0]), silu_f(v1[1])), by = cvt_pk_bf16(silu_f(v1[2]), silu_f(v1[3]));
                    const u32x2 sx = __builtin_amdgcn_permlane16_swap(ax, bx, false, false), sy = __builtin_amdgcn_permlane16_swap(ay, by, false, false);
                    u32x4 w; w.x = sx.x; w.y = sy.x; w.z = sx.y; w.w = sy.y;
                    if (bad) { w.x = 0x7fc07fc0u; w.y = 0x7fc07fc0u; w.z = 0x7fc07fc0u; w.w = 0x7fc07fc0u; }
                    *(u32x4*)(rowp + bj * HALF) = w; } }
    }
    if (tid == 0) __hip_atomic_fetch_add(ctr, 1u, __ATOMIC_RELAXED, __HIP_MEMORY_SCOPE_AGENT);
    WAIT_V(0);
}
}

extern "C" void kernel_launch(int rank, int nranks, ncclComm_t comm,
                              hipStream_t stream,
                              void* const* d_in, const int* in_sizes, int n_in,
                              void* d_out, int out_size,
                              const int* in_dtypes, int out_dtype,
                              void* const* d_out_all,
                              void* const* d_ws_all, size_t ws_bytes) {
    using namespace agg;
    static bool init = false;
    if (!init) { (void)hipFuncSetAttribute((const void*)ag_gemm_k, hipFuncAttributeMaxDynamicSharedMemorySize, STAGE_BYTES); init = true; }
    if (nranks != NR || ws_bytes < WS_NEED) { fprintf(stderr, "unsupported configuration\n"); return; }
    Params p;
    p.x = (const bf16_t*)d_in[0]; p.w = (const bf16_t*)d_in[1]; p.out = (bf16_t*)d_out; p.rank = rank;
    for (int i = 0; i < NR; ++i) p.ws[i] = (unsigned char*)d_ws_all[i];
    ag_gemm_k<<<NWG, 512, STAGE_BYTES, stream>>>(p);
}
```

```cpp
#include <hip/hip_runtime.h>
#include <rccl/rccl.h>
#include <cstdio>

namespace agg {
#define LAS __attribute__((address_space(3)))
typedef unsigned short bf16_t;
typedef short bf16x8 __attribute__((ext_vector_type(8)));
typedef short s16x4 __attribute__((ext_vector_type(4)));
typedef float f32x4 __attribute__((ext_vector_type(4)));
typedef unsigned u32x4 __attribute__((ext_vector_type(4)));
typedef unsigned u32x2 __attribute__((ext_vector_type(2)));

constexpr int NR = 8;
constexpr int MM = 4096, NN = 4096, KK = 4096, MSH = MM / NR;
constexpr int BM = 256, BK = 64, HALF = 128, HTB = HALF * BK * 2, GEMM_LDS = 8 * HTB, NXCD = 8, WGM = 8;
constexpr int XBUF = GEMM_LDS, PBUF = GEMM_LDS + 4096, PFBUF = GEMM_LDS + 4096 + 256, STAGE_BYTES = GEMM_LDS + 4096 + 512;
constexpr int NWG = 256;
constexpr int NCH = 32;
#define AGG_D 8
constexpr int DLA = AGG_D;
constexpr size_t GATHER_BYTES = (size_t)MM * KK * 2;
constexpr size_t FLAG_OFF = 2 * GATHER_BYTES;
constexpr size_t CTL_OFF = FLAG_OFF + 64 * 1024;
constexpr size_t WS_NEED = CTL_OFF + 4096;
constexpr unsigned long long DEADLINE_TICKS = 50000000ull;

template <int V> struct IC { static constexpr int value = V; };
struct Params { const bf16_t* x; const bf16_t* w; bf16_t* out; unsigned char* ws[NR]; int rank; };

__host__ __device__ __forceinline__ int lds_byte(int r, int c) { const int st = (r >> 4) * 2 + (c >> 5), rr = r & 15, cc = c & 31, ob = rr * 64 + cc * 2; return st * 1024 + (ob ^ (((ob >> 9) & 1) << 5)); }
__host__ __device__ __forceinline__ void stage_rc(int b, int& R, int& C) { const int st = b / 1024, sb = b % 1024, swz = sb ^ (((sb >> 9) & 1) << 5); R = (st >> 1) * 16 + swz / 64; C = (st & 1) * 32 + (swz % 64) / 2; }
__host__ __device__ __forceinline__ int bswz(int row) { return ((row & 3) | (((row >> 3) & 1) << 2)) << 1; }

__device__ __forceinline__ unsigned cvt_pk_bf16(float lo, float hi) { unsigned r; asm volatile("v_cvt_pk_bf16_f32 %0, %1, %2" : "=v"(r) : "v"(lo), "v"(hi)); return r; }
__device__ __forceinline__ float silu_f(float y) {
    const float c = fminf(fmaxf(y, -60.f), 60.f);
    const float e = __builtin_amdgcn_exp2f(c * -1.44269504088896340736f);
    return y * __builtin_amdgcn_rcpf(1.0f + e);
}

__global__ __launch_bounds__(512, 2) void ag_gemm_k(const Params P) {
    extern __shared__ __attribute__((aligned(16))) unsigned char shm[];
    LAS unsigned char* lds = (LAS unsigned char*)shm;
    const int tid = threadIdx.x, wid = __builtin_amdgcn_readfirstlane(tid >> 6), lane = tid & 63, wr = wid >> 2, wc = wid & 3, fr = lane & 15, fq = lane >> 4;
    constexpr int K = KK, N = NN, nt = K / BK, nN = NN / BM;
    int wgid = blockIdx.x; { const int q = NWG / NXCD, xcd = wgid % NXCD, off = wgid / NXCD; wgid = xcd * q + off; }
    const int nig = WGM * nN, gid = wgid / nig, fm = gid * WGM;
    const int pm = fm + ((wgid % nig) % WGM), pn = (wgid % nig) / WGM;
    const int rank = P.rank, src = pm >> 1;
    const bool remote = (src != rank);
    unsigned char* const myws = P.ws[rank];
    const bool comm = (wid == 7);
#define AGG_H2 2
#define AGG_W1 2
    const bool helper = (wid == 3 || wid == AGG_H2);
    const bool poller = (wid == 4);

#define SA(b, h) (((b) * 2 + (h)) * HTB)
#define SB(b, h) ((4 + (b) * 2 + (h)) * HTB)
    unsigned voffA[2], voffB;
#pragma unroll
    for (int i = 0; i < 2; ++i) { int R, C; stage_rc(tid * 16 + i * 8192, R, C); voffA[i] = (unsigned)(R * K + C) * 2u; }
    { const int b = tid * 16, row = b >> 8, chp = (b >> 4) & 15, ch = chp ^ bswz(row); voffB = (unsigned)(row * N * 2 + ch * 16); }
    const size_t kstepA = (size_t)(BK * 2), kstepB = (size_t)BK * N * 2;
    const size_t hstepA = (size_t)HALF * K * 2, hstepB = (size_t)HALF * 2;
    const unsigned ldsw = (unsigned)wid * 1024u;
    const unsigned hA = (wid == 3) ? 32u * K * 2u : (7u - (AGG_H2 >> 1)) * 16u * K * 2u + ((AGG_H2 & 1) ? 0u : 64u);
    const unsigned hB = (wid == 3) ? 16u * N * 2u : (15u - AGG_H2) * 4u * N * 2u;
    const unsigned hL = (wid == 3) ? 7u * 1024u : 15u * 1024u;
    const int aoff = lds_byte(wr * 64 + fr, fq * 8);
    int boff[2];
    { const int g = fq, q = (lane & 15) >> 2, p = lane & 3, row0 = 8 * g + q;
#pragma unroll
      for (int n = 0; n < 2; ++n) { const int c = 2 * wc + n; boff[n] = (int)(unsigned)(size_t)lds + SB(0, 0) + 256 * row0 + 16 * ((2 * c + (p >> 1)) ^ bswz(row0)) + 8 * (p & 1); } }
#define GLDS(g, l) __builtin_amdgcn_global_load_lds((const unsigned*)(g), (LAS unsigned*)(l), 16, 0, 0)
#define STAGE_A(bufoff, gbase) do { if (!comm) { _Pragma("unroll") for (int _i = 0; _i < 2; ++_i) GLDS((const char*)(gbase) + voffA[_i], lds + (bufoff) + ldsw + _i * 8192); } } while (0)
#define STAGE_B(bufoff, gbase) do { if (!comm) { _Pragma("unroll") for (int _i = 0; _i < 2; ++_i) GLDS((const char*)(gbase) + (size_t)_i * 32 * N * 2 + voffB, lds + (bufoff) + ldsw + _i * 8192); } } while (0)
#define HELP_A(bufoff, gbase) do { if (helper) GLDS((const char*)(gbase) + hA + voffA[0], lds + (bufoff) + hL); } while (0)
#define HELP_B(bufoff, gbase) do { if (helper) GLDS((const char*)(gbase) + hB + voffB, lds + (bufoff) + hL); } while (0)
#define LDA(dst, b, h) do { _Pragma("unroll") for (int m = 0; m < 4; ++m) _Pragma("unroll") for (int k = 0; k < 2; ++k) dst[m][k] = *(const LAS bf16x8*)(lds + SA(b, h) + aoff + m * 2048 + k * 1024); } while (0)
#define TRRD(dst, addr, off) asm volatile("ds_read_b64_tr_b16 %0, %1 offset:%2" : "=v"(dst) : "v"(addr), "i"(off) : "memory")
#define LDB(dst, b, h) do { _Pragma("unroll") for (int n = 0; n < 2; ++n) _Pragma("unroll") for (int k = 0; k < 2; ++k) { \
        s16x4 lo, hi; TRRD(lo, boff[n], ((b) * 2 + (h)) * HTB + k * 8192); TRRD(hi, boff[n], ((b) * 2 + (h)) * HTB + k * 8192 + 1024); \
        dst[n][k] = __builtin_shufflevector(lo, hi, 0, 1, 2, 3, 4, 5, 6, 7); } } while (0)
#define MMA_K(ai, bj, At, Bt, k) do { _Pragma("unroll") for (int m = 0; m < 4; ++m) _Pragma("unroll") for (int n = 0; n < 2; ++n) \
        acc[ai][bj][m][n] = __builtin_amdgcn_mfma_f32_16x16x32_bf16(Bt[n][k], At[m][k], acc[ai][bj][m][n], 0, 0, 0); } while (0)
#define MMA(ai, bj, At, Bt, MID) do { __builtin_amdgcn_s_setprio(1); MMA_K(ai, bj, At, Bt, 0); SCHED; MID; SCHED; MMA_K(ai, bj, At, Bt, 1); __builtin_amdgcn_s_setprio(0); } while (0)
#define WAIT_V(n) asm volatile("s_waitcnt vmcnt(" #n ")" ::: "memory")
#define WAIT_VC(n2, n3) do { if (!comm) { if (helper) WAIT_V(n3); else WAIT_V(n2); } } while (0)
#define WAIT_L(n) asm volatile("s_waitcnt lgkmcnt(" #n ")" ::: "memory")
#define BAR __builtin_amdgcn_s_barrier()
#define SCHED __builtin_amdgcn_sched_barrier(0)

#define STAMP() do {} while (0)
#define STAMP2(i) do {} while (0)
    STAMP();
    f32x4 acc[2][2][4][2];
#pragma unroll
    for (int a = 0; a < 2; ++a)
#pragma unroll
        for (int b = 0; b < 2; ++b)
#pragma unroll
            for (int m = 0; m < 4; ++m)
#pragma unroll
                for (int n = 0; n < 2; ++n) acc[a][b][m][n] = (f32x4){0.f, 0.f, 0.f, 0.f};
    bf16x8 At[4][2], B0[2][2], B1[2][2];
    const char* cB = (const char*)P.w + (size_t)pn * 2 * hstepB;

#define AGG_PD 3
    const unsigned pfo = (unsigned)((((int)blockIdx.x >> 3) * 4 + (lane >> 4)) * N * 2 + (pn & ~3) * 512 + (lane & 15) * 128);
#define PFW(i) __builtin_amdgcn_global_load_lds((const unsigned*)((const char*)P.w + (size_t)(i) * 128 * N * 2 + pfo), (LAS unsigned*)(lds + PFBUF), 4, 0, 0)
    const int pi = (int)blockIdx.x >> 1, ph = (int)blockIdx.x & 1;
    const int npeer = ph ? 3 : 4;
    const unsigned xo0 = (unsigned)((4 * pi + (lane >> 4)) * K * 2 + (lane & 15) * 16);
    unsigned xo = xo0;
    const char* xsrc = (const char*)P.x + xo0;
    unsigned* const ctr = (unsigned*)(myws + CTL_OFF);
    unsigned* const tmo = (unsigned*)(myws + CTL_OFF + 256);
#define XLOAD(c) __builtin_amdgcn_global_load_lds((const unsigned*)(xsrc + (size_t)(c) * 256), (LAS unsigned*)(lds + XBUF + ((c) & 3) * 1024), 16, 0, 0)

    unsigned cv;
    asm volatile("s_nop 4\n\tglobal_load_dword %0, %1, %2 sc1" : "=v"(cv) : "v"(0u), "s"(ctr) : "memory");
    if (comm) { XLOAD(0); XLOAD(1); XLOAD(2); WAIT_V(0); }
    else if (wid == 6) {
        WAIT_V(0);
    }
    else if (wid < AGG_PD) { PFW(wid + 1); WAIT_V(1); }
    else WAIT_V(0);
    asm volatile("" : "+v"(cv));
    STAMP();
    const unsigned call = (unsigned)__builtin_amdgcn_readfirstlane(cv) / NWG + 1u;
    const unsigned fbase = call * 64u;
    const size_t par_off = (size_t)(call & 1u) * GATHER_BYTES;
    const size_t slot_off = par_off + (size_t)rank * MSH * K * 2;
    const int rot = ph ? pi % 3 : pi & 3;
    const char* pd[4];
#pragma unroll
    for (int k = 0; k < 4; ++k) { int q = k + rot; if (q >= npeer) q -= npeer; if (q >= npeer) q = 0;
        pd[k] = (const char*)P.ws[(rank + 1 + ph * 4 + q) & (NR - 1)] + slot_off; }
    const char* pflag;
    { const char* b = (const char*)P.ws[(rank + 1 + ph * 4) & (NR - 1)];
#pragma unroll
      for (int i = 1; i < 4; ++i) b = (lane == i && i < npeer) ? (const char*)P.ws[(rank + 1 + ph * 4 + i) & (NR - 1)] : b;
      pflag = b + FLAG_OFF + (size_t)rank * 512 + (size_t)pi * 4; }
    const unsigned long long fmask = ph ? 7ull : 15ull;
    const char* qflag = (const char*)myws + FLAG_OFF + (size_t)src * 512 + (size_t)(pm & 1) * 256 + (size_t)lane * 4;
    const unsigned lds0 = (unsigned)(size_t)lds;
    u32x4 xv = (u32x4){0u, 0u, 0u, 0u};
    unsigned pf = 0, tnow = 0;

    unsigned e_iss = 0, e_i0 = 0, e_i1 = 0, e_due = 0;
#define AGG_TAU 275u
    int e_np = 0, e_r = 0, e_ack = 0, e_have = (remote && (comm || poller)) ? 0 : 1000;
    int e_act = 0, timed_out = 0;
#define VMCNT_NOW(dst) do { unsigned _v; asm volatile("s_getreg_b32 %0, hwreg(HW_REG_IB_STS, 0, 24)" : "=s"(_v) :: "memory"); dst = (_v & 15u) | ((_v >> 18) & 48u); } while (0)
#define DECIDE(lim, now) do { unsigned _c; VMCNT_NOW(_c); const unsigned _done = e_iss - _c; \
        if (e_np > 0 && (int)(_done - e_i0) >= 0) { STAMP2(e_ack); ++e_ack; --e_np; e_i0 = e_i1; const unsigned _fv = fbase + (unsigned)e_ack; \
            asm volatile("s_mov_b64 exec, %2\n\tglobal_store_dword %0, %1, off sc0 sc1\n\ts_mov_b64 exec, -1" :: "v"(pflag), "v"(_fv), "s"(fmask) : "memory"); ++e_iss; } \
        e_act = (int)((((unsigned)(e_r - (lim))) & ((unsigned)(e_np - 2)) & ~((now) - e_due)) >> 31); \
    } while (0)
#define NOW() ((unsigned)__builtin_amdgcn_s_memrealtime())
#define PSLEEP do {} while (0)
#define POLL() __builtin_amdgcn_global_load_lds((const unsigned*)qflag, (LAS unsigned*)(lds + PBUF), 4, 0, 17)
#define PREAD() asm volatile("ds_read_b32 %0, %1" : "=v"(pf) : "v"(lds0 + PBUF + (unsigned)lane * 4u) : "memory")
#define PEVAL() do { asm volatile("" : "+v"(pf)); const unsigned _n1 = fbase + (unsigned)e_have + 1u; \
        if (__all(pf >= _n1)) { STAMP2(32 + e_have); ++e_have; if (__all(pf > _n1)) { STAMP2(32 + e_have); ++e_have; } } } while (0)
#define EVAL() do { PREAD(); WAIT_L(0); PEVAL(); } while (0)
#define XREAD(c) asm volatile("ds_read_b128 %0, %1" : "=v"(xv) : "v"(lds0 + XBUF + (unsigned)((c) & 3) * 1024u + (unsigned)lane * 16u) : "memory")
#define PST(k) do { asm volatile("s_nop 4\n\tglobal_store_dwordx4 %0, %1, %2 sc0 sc1\n\ts_nop 1" :: "v"(xo), "v"(xv), "s"(pd[k]) : "memory"); ++e_iss; } while (0)
#define SLOT(k) do { asm volatile("" : "+s"(e_act)); if (e_act != 0) { \
        if ((k) == 0 && e_r + 3 < NCH) { XLOAD(e_r + 3); ++e_iss; } \
        if ((k) < 3 || npeer == 4) PST(k); \
        if ((k) == 3) { if (e_np == 0) e_i0 = e_iss; e_i1 = e_iss; ++e_np; ++e_r; e_due += (e_r <= AGG_N1) ? AGG_TAU1 : AGG_TAU; xo += 256u; e_act = 0; XREAD(e_r); } } } while (0)
#define TICK(lim) do { DECIDE(lim, NOW()); WAIT_L(0); SLOT(0); SLOT(1); SLOT(2); SLOT(3); } while (0)
#define BOUNDED_UNTIL(cond, BODY) do { if (!(cond)) { const unsigned long long _t0 = __builtin_amdgcn_s_memrealtime(); unsigned _sp = 0; \
        while (!(cond)) { BODY; \
            if ((++_sp & 63u) == 0u) { if (__builtin_amdgcn_s_memrealtime() - _t0 > DEADLINE_TICKS) { timed_out = 1; break; } } \
            __builtin_amdgcn_s_sleep(1); } \
        if (timed_out != 0) { if (lane == 0) asm volatile("s_nop 4\n\tglobal_store_dword %0, %1, %2 sc1\n\ts_waitcnt vmcnt(0)" :: "v"(0u), "v"(call), "s"(tmo) : "memory"); ++e_iss; e_have = 1000; } } } while (0)
#define GATE(need, lim) do { if (e_have < (need)) { \
        if (GROLE == 3) { SLOT(2); SLOT(3); BOUNDED_UNTIL(e_have >= (need), { EVAL(); TICK(lim); }); } \
        else { BOUNDED_UNTIL(e_have >= (need), { POLL(); WAIT_V(0); EVAL(); PSLEEP; }); } } } while (0)

    if (poller) { asm volatile("ds_write_b32 %0, %1\n\ts_waitcnt lgkmcnt(0)" :: "v"(lds0 + PBUF + (unsigned)lane * 4u), "v"(0u) : "memory"); }
#define AGG_N1 0
#define AGG_TAU1 AGG_TAU
#define AGG_PHI 256
    e_due = (unsigned)__builtin_amdgcn_s_memrealtime() + (unsigned)blockIdx.x * AGG_TAU1 * AGG_PHI / 65536u;
    if (comm) { XREAD(0); TICK(DLA); }
    BAR;
    STAGE_B(SB(0, 0), cB); HELP_B(SB(0, 0), cB); STAGE_B(SB(0, 1), cB + hstepB); HELP_B(SB(0, 1), cB + hstepB);
    STAGE_B(SB(1, 0), cB + kstepB); HELP_B(SB(1, 0), cB + kstepB); STAGE_B(SB(1, 1), cB + hstepB + kstepB); HELP_B(SB(1, 1), cB + hstepB + kstepB);
#define GROLE (comm ? 3 : 2)
    GATE(1, DLA);
#undef GROLE
#undef GATE
#define GATE(need, lim) do { if (e_have < (need)) { SLOT(2); SLOT(3); unsigned _tp = NOW() - 1000u; \
        BOUNDED_UNTIL(e_have >= (need), { EVAL(); if (e_have < (need)) { if (NOW() - _tp > 40u) { POLL(); ++e_iss; _tp = NOW(); } TICK(lim); } }); } } while (0)
#define AGG_LW 2
    if (comm && !remote) BOUNDED_UNTIL(e_ack >= AGG_LW, TICK(DLA));
    STAMP();
    const char* cA = remote ? (const char*)myws + par_off + (size_t)pm * 2 * hstepA : (const char*)P.x + (size_t)(pm & 1) * 2 * hstepA;
    BAR;
    STAGE_A(SA(0, 0), cA); HELP_A(SA(0, 0), cA); STAGE_A(SA(0, 1), cA + hstepA); HELP_A(SA(0, 1), cA + hstepA); STAGE_A(SA(1, 0), cA + kstepA); HELP_A(SA(1, 0), cA + kstepA);
    if (comm) TICK(DLA);
    if (poller && remote) { POLL(); WAIT_V(3); } else WAIT_VC(2, 3);
    BAR;
    if (comm) { tnow = NOW(); }
    if (wr == 1) BAR;
    STAMP();
#define RSTAGE_A(bufoff, gbase) do { if (ROLE != 3) { _Pragma("unroll") for (int _i = 0; _i < 2; ++_i) GLDS((const char*)(gbase) + voffA[_i], lds + (bufoff) + ldsw + _i * 8192); } } while (0)
#define RSTAGE_B(bufoff, gbase) do { if (ROLE != 3) { _Pragma("unroll") for (int _i = 0; _i < 2; ++_i) GLDS((const char*)(gbase) + (size_t)_i * 32 * N * 2 + voffB, lds + (bufoff) + ldsw + _i * 8192); } } while (0)
#define RHELP_A(bufoff, gbase) do { if (ROLE == 1) GLDS((const char*)(gbase) + hA + voffA[0], lds + (bufoff) + hL); } while (0)
#define RHELP_B(bufoff, gbase) do { if (ROLE == 1) GLDS((const char*)(gbase) + hB + voffB, lds + (bufoff) + hL); } while (0)
#define RWAIT() do { if (ROLE == 1) WAIT_V(8); else if (ROLE != 3) WAIT_V(6); } while (0)
#define RSLOT(k) do { if (ROLE == 3) SLOT(k); } while (0)
    auto kloop = [&](auto role_c, const int t_beg, const int t_end) __attribute__((always_inline)) {
      constexpr bool EAGER = decltype(role_c)::value == 4;
      constexpr int ROLE = EAGER ? 3 : decltype(role_c)::value;
      for (int t = t_beg; t < t_end; t += 2) {
        const int it = t >> 1;
        const bool last = (t == nt - 2);
        const char* a1 = cA + (size_t)(t + 1) * kstepA;
        const int t2 = last ? 0 : t + 2;
        const char* a2 = cA + (size_t)t2 * kstepA; const char* b2 = cB + (size_t)t2 * kstepB;
        const char* a3 = a2 + kstepA; const char* b3 = b2 + kstepB;
        const int want = (it + 4 < NCH) ? it + 4 : NCH;
        const int need = (it + 2 < NCH) ? it + 2 : NCH;
        const int lim = (it + DLA < NCH) ? it + DLA : NCH;
        const int lim8 = (it + 1 + DLA < NCH) ? it + 1 + DLA : NCH;
#define AGG_VA 0
#define AGG_VB 1
#define ASLOT(k) do { if (ROLE == 3 && !EAGER) SLOT(k); } while (0)
#define ETICK() do { if (EAGER) { WAIT_L(0); unsigned _c2; VMCNT_NOW(_c2); \
            const int _y = e_np > 0 ? (int)(e_iss - _c2 - e_i0) : -1; \
            const int _x = (int)((tnow - e_due) | ~(((unsigned)(e_r - lim)) & ((unsigned)(e_np - 2)))); \
            if ((_x & _y) >= 0) { DECIDE(lim, tnow); SLOT(0); SLOT(1); SLOT(2); SLOT(3); } \
            tnow = NOW(); } } while (0)
        ETICK();
        if (ROLE == 3) WAIT_L(0);
        if (ROLE == 3 && !AGG_VB) PREAD();
        LDB(B0, 0, 0); SCHED; LDA(At, 0, 0); RSTAGE_A(SA(1, 1), a1 + hstepA);
        ASLOT(0);
        WAIT_L(8); BAR; WAIT_L(0); SCHED; MMA(0, 0, At, B0, RHELP_A(SA(1, 1), a1 + hstepA)); BAR; SCHED;
        ETICK();
        if (ROLE == 3 && AGG_VB) PREAD();
        LDB(B1, 0, 1); RSTAGE_B(SB(0, 0), b2);
        ASLOT(1);
        if (ROLE == 3) { if (AGG_VB) WAIT_L(8); if (e_have < want) PEVAL();
          GATE(need, lim);
        }
        BAR; WAIT_L(0); SCHED; MMA(0, 1, At, B1, RHELP_B(SB(0, 0), b2)); BAR;
        ETICK();
        LDA(At, 0, 1); RSTAGE_A(SA(0, 0), a2);
        ASLOT(2); if (ROLE == 3 && !EAGER) tnow = NOW();
        BAR; WAIT_L(0); SCHED; MMA(1, 0, At, B0, RHELP_A(SA(0, 0), a2)); BAR; SCHED;
        ETICK();
        RSTAGE_B(SB(0, 1), b2 + hstepB);
        ASLOT(3); if (ROLE == 3 && !EAGER) DECIDE(lim, tnow);
        RWAIT();
        BAR; MMA(1, 1, At, B1, RHELP_B(SB(0, 1), b2 + hstepB)); BAR;
        ETICK();
        if (ROLE == 3) WAIT_L(0);
        LDB(B0, 1, 0); SCHED; LDA(At, 1, 0); RSTAGE_A(SA(0, 1), a2 + hstepA);
        ASLOT(0); if (ROLE == 3 && !AGG_VB) { if (e_have < want) { POLL(); ++e_iss; } }
        WAIT_L(8); BAR; WAIT_L(0); SCHED; MMA(0, 0, At, B0, RHELP_A(SA(0, 1), a2 + hstepA)); BAR; SCHED;
        ETICK();
        LDB(B1, 1, 1); RSTAGE_B(SB(1, 0), b3);
        ASLOT(1);
        if (ROLE == 3) { if (it + 1 + AGG_PD < NCH) { PFW(it + 1 + AGG_PD); ++e_iss; } }
        BAR; WAIT_L(0); SCHED; MMA(0, 1, At, B1, RHELP_B(SB(1, 0), b3)); BAR;
        ETICK();
        LDA(At, 1, 1); RSTAGE_A(SA(1, 0), a3);
        ASLOT(2); if (ROLE == 3) { if (AGG_VB) { if (e_have < want + 1) { POLL(); ++e_iss; } } if (!EAGER) tnow = NOW(); }
        BAR; WAIT_L(0); SCHED; MMA(1, 0, At, B0, RHELP_A(SA(1, 0), a3)); BAR; SCHED;
        ETICK();
        RSTAGE_B(SB(1, 1), b3 + hstepB);
        ASLOT(3); if (ROLE == 3 && !EAGER) DECIDE(lim8, tnow);
        RWAIT();
        BAR; MMA(1, 1, At, B1, RHELP_B(SB(1, 1), b3 + hstepB)); BAR;
        STAMP();
      }
    };
#define AGG_NE 4
    if (comm) { kloop(IC<4>{}, 0, 2 * AGG_NE); kloop(IC<3>{}, 2 * AGG_NE, nt); } else if (helper) kloop(IC<1>{}, 0, nt); else kloop(IC<0>{}, 0, nt);
    if (wr == 0) BAR;
    if (comm) { WAIT_L(0); SLOT(0); SLOT(1); SLOT(2); SLOT(3); BOUNDED_UNTIL(e_ack >= NCH, TICK(NCH)); }
    {
        unsigned tv; asm volatile("s_nop 4\n\tglobal_load_dword %0, %1, %2 sc1\n\ts_waitcnt vmcnt(0)" : "=v"(tv) : "v"(0u), "s"(tmo) : "memory");
        const bool bad = ((unsigned)__builtin_amdgcn_readfirstlane(tv) == call);
        const int row0 = pm * BM + wr * 64 + fr, col0 = pn * BM + wc * 32 + (fq & 1) * 16 + (fq >> 1) * 8;
#pragma unroll
        for (int ai = 0; ai < 2; ++ai)
#pragma unroll
            for (int m = 0; m < 4; ++m) { bf16_t* rowp = P.out + (size_t)(row0 + ai * HALF + m * 16) * N + col0;
#pragma unroll
                for (int bj = 0; bj < 2; ++bj) { const f32x4 v0 = acc[ai][bj][m][0], v1 = acc[ai][bj][m][1];
                    const unsigned ax = cvt_pk_bf16(silu_f(v0[0]), silu_f(v0[1])), ay = cvt_pk_bf16(silu_f(v0[2]), silu_f(v0[3]));
                    const unsigned bx = cvt_pk_bf16(silu_f(v1[0]), silu_f(v1[1])), by = cvt_pk_bf16(silu_f(v1[2]), silu_f(v1[3]));
                    const u32x2 sx = __builtin_amdgcn_permlane16_swap(ax, bx, false, false), sy = __builtin_amdgcn_permlane16_swap(ay, by, false, false);
                    u32x4 w; w.x = sx.x; w.y = sy.x; w.z = sx.y; w.w = sy.y;
                    if (bad) { w.x = 0x7fc07fc0u; w.y = 0x7fc07fc0u; w.z = 0x7fc07fc0u; w.w = 0x7fc07fc0u; }
                    *(u32x4*)(rowp + bj * HALF) = w; } }
    }
    if (tid == 0) __hip_atomic_fetch_add(ctr, 1u, __ATOMIC_RELAXED, __HIP_MEMORY_SCOPE_AGENT);
    WAIT_V(0);
}
}

extern "C" void kernel_launch(int rank, int nranks, ncclComm_t comm,
                              hipStream_t stream,
                              void* const* d_in, const int* in_sizes, int n_in,
                              void* d_out, int out_size,
                              const int* in_dtypes, int out_dtype,
                              void* const* d_out_all,
                              void* const* d_ws_all, size_t ws_bytes) {
    using namespace agg;
    static bool init = false;
    if (!init) { (void)hipFuncSetAttribute((const void*)ag_gemm_k, hipFuncAttributeMaxDynamicSharedMemorySize, STAGE_BYTES); init = true; }
    if (nranks != NR || ws_bytes < WS_NEED) { fprintf(stderr, "unsupported configuration\n"); return; }
    Params p;
    p.x = (const bf16_t*)d_in[0]; p.w = (const bf16_t*)d_in[1]; p.out = (bf16_t*)d_out; p.rank = rank;
    for (int i = 0; i < NR; ++i) p.ws[i] = (unsigned char*)d_ws_all[i];
    ag_gemm_k<<<NWG, 512, STAGE_BYTES, stream>>>(p);
}
```

```cpp
#include <hip/hip_runtime.h>
#include <rccl/rccl.h>
#include <cstdio>

namespace agg {
#define LAS __attribute__((address_space(3)))
typedef unsigned short bf16_t;
typedef short bf16x8 __attribute__((ext_vector_type(8)));
typedef short s16x4 __attribute__((ext_vector_type(4)));
typedef float f32x4 __attribute__((ext_vector_type(4)));
typedef unsigned u32x4 __attribute__((ext_vector_type(4)));
typedef unsigned u32x2 __attribute__((ext_vector_type(2)));

constexpr int NR = 8;
constexpr int MM = 4096, NN = 4096, KK = 4096, MSH = MM / NR;
constexpr int BM = 256, BK = 64, HALF = 128, HTB = HALF * BK * 2, GEMM_LDS = 8 * HTB, NXCD = 8, WGM = 8;
constexpr int XBUF = GEMM_LDS, PBUF = GEMM_LDS + 4096, PFBUF = GEMM_LDS + 4096 + 256, TBUF = GEMM_LDS + 4096 + 512, TFLAG = TBUF + 256, STAGE_BYTES = TFLAG + 64;
constexpr int NWG = 256;
constexpr int NCH = 32;
#define AGG_D 8
constexpr int DLA = AGG_D;
#define AGG_RSR 8448
constexpr unsigned RSR = AGG_RSR;
constexpr size_t GATHER_BYTES = (size_t)MM * RSR;
constexpr size_t FLAG_OFF = 2 * GATHER_BYTES;
constexpr size_t CTL_OFF = FLAG_OFF + 64 * 1024;
constexpr size_t WS_NEED = CTL_OFF + 4096;
constexpr unsigned long long DEADLINE_TICKS = 50000000ull;

template <int V> struct IC { static constexpr int value = V; };
struct Params { const bf16_t* x; const bf16_t* w; bf16_t* out; unsigned char* ws[NR]; int rank; };

__host__ __device__ __forceinline__ int lds_byte(int r, int c) { const int st = (r >> 4) * 2 + (c >> 5), rr = r & 15, cc = c & 31, ob = rr * 64 + cc * 2; return st * 1024 + (ob ^ (((ob >> 9) & 1) << 5)); }
__host__ __device__ __forceinline__ void stage_rc(int b, int& R, int& C) { const int st = b / 1024, sb = b % 1024, swz = sb ^ (((sb >> 9) & 1) << 5); R = (st >> 1) * 16 + swz / 64; C = (st & 1) * 32 + (swz % 64) / 2; }
__host__ __device__ __forceinline__ int bswz(int row) { return ((row & 3) | (((row >> 3) & 1) << 2)) << 1; }

__device__ __forceinline__ unsigned cvt_pk_bf16(float lo, float hi) { unsigned r; asm volatile("v_cvt_pk_bf16_f32 %0, %1, %2" : "=v"(r) : "v"(lo), "v"(hi)); return r; }
typedef float f32x2 __attribute__((ext_vector_type(2)));
__device__ __forceinline__ unsigned silu2_bf16(float y0, float y1) {
    constexpr float L2E = 1.44269504088896340736f, LIM = 60.f * L2E;
    const f32x2 y = {y0, y1};
    f32x2 t = y * (f32x2){-L2E, -L2E};
    f32x2 e = {__builtin_amdgcn_exp2f(__builtin_amdgcn_fmed3f(t.x, -LIM, LIM)), __builtin_amdgcn_exp2f(__builtin_amdgcn_fmed3f(t.y, -LIM, LIM))};
    e = e + (f32x2){1.f, 1.f};
    const f32x2 r = {__builtin_amdgcn_rcpf(e.x), __builtin_amdgcn_rcpf(e.y)};
    const f32x2 o = y * r;
    return cvt_pk_bf16(o.x, o.y);
}

__global__ __launch_bounds__(512, 2) void ag_gemm_k(const Params P) {
    extern __shared__ __attribute__((aligned(16))) unsigned char shm[];
    LAS unsigned char* lds = (LAS unsigned char*)shm;
    const int tid = threadIdx.x, wid = __builtin_amdgcn_readfirstlane(tid >> 6), lane = tid & 63, wr = wid >> 2, wc = wid & 3, fr = lane & 15, fq = lane >> 4;
    constexpr int K = KK, N = NN, nt = K / BK, nN = NN / BM;
    int wgid = blockIdx.x; { const int q = NWG / NXCD, xcd = wgid % NXCD, off = wgid / NXCD; wgid = xcd * q + off; }
    const int nig = WGM * nN, gid = wgid / nig, fm = gid * WGM;
    const int pm = fm + ((wgid % nig) % WGM), pn = (wgid % nig) / WGM;
    const int rank = P.rank, src = pm >> 1;
    const bool remote = (src != rank);
    unsigned char* const myws = P.ws[rank];
    const bool comm = (wid == 7);
#define AGG_H2 2
#define AGG_W1 2
    const bool helper = (wid == 3 || wid == AGG_H2);
    const bool poller = (wid == 4);

#define SA(b, h) (((b) * 2 + (h)) * HTB)
#define SB(b, h) ((4 + (b) * 2 + (h)) * HTB)
    unsigned voffA[2], voffB;
    const unsigned RS = remote ? RSR : (unsigned)K * 2u; constexpr unsigned CS = 256u;
#pragma unroll
    for (int i = 0; i < 2; ++i) { int R, C; stage_rc(tid * 16 + i * 8192, R, C); voffA[i] = (unsigned)R * RS + (unsigned)C * 2u; }
    { const int b = tid * 16, row = b >> 8, chp = (b >> 4) & 15, ch = chp ^ bswz(row); voffB = (unsigned)(row * N * 2 + ch * 16); }
    const size_t kstepA = (size_t)(BK * 2), kstepB = (size_t)BK * N * 2;
    const size_t hstepA = (size_t)HALF * RS, hstepB = (size_t)HALF * 2;
    const unsigned ldsw = (unsigned)wid * 1024u;
    const unsigned hA = (wid == 3) ? 32u * RS : (7u - (AGG_H2 >> 1)) * 16u * RS + ((AGG_H2 & 1) ? 0u : 64u);
    const unsigned hB = (wid == 3) ? 16u * N * 2u : (15u - AGG_H2) * 4u * N * 2u;
    const unsigned hL = (wid == 3) ? 7u * 1024u : 15u * 1024u;
    const int aoff = lds_byte(wr * 64 + fr, fq * 8);
    int boff[2];
    { const int g = fq, q = (lane & 15) >> 2, p = lane & 3, row0 = 8 * g + q;
#pragma unroll
      for (int n = 0; n < 2; ++n) { const int c = 2 * wc + n; boff[n] = (int)(unsigned)(size_t)lds + SB(0, 0) + 256 * row0 + 16 * ((2 * c + (p >> 1)) ^ bswz(row0)) + 8 * (p & 1); } }
#define GLDS(g, l) __builtin_amdgcn_global_load_lds((const unsigned*)(g), (LAS unsigned*)(l), 16, 0, 0)
#define STAGE_A(bufoff, gbase) do { if (!comm) { _Pragma("unroll") for (int _i = 0; _i < 2; ++_i) GLDS((const char*)(gbase) + voffA[_i], lds + (bufoff) + ldsw + _i * 8192); } } while (0)
#define STAGE_B(bufoff, gbase) do { if (!comm) { _Pragma("unroll") for (int _i = 0; _i < 2; ++_i) GLDS((const char*)(gbase) + (size_t)_i * 32 * N * 2 + voffB, lds + (bufoff) + ldsw + _i * 8192); } } while (0)
#define HELP_A(bufoff, gbase) do { if (helper) GLDS((const char*)(gbase) + hA + voffA[0], lds + (bufoff) + hL); } while (0)
#define HELP_B(bufoff, gbase) do { if (helper) GLDS((const char*)(gbase) + hB + voffB, lds + (bufoff) + hL); } while (0)
#define LDA(dst, b, h) do { _Pragma("unroll") for (int m = 0; m < 4; ++m) _Pragma("unroll") for (int k = 0; k < 2; ++k) dst[m][k] = *(const LAS bf16x8*)(lds + SA(b, h) + aoff + m * 2048 + k * 1024); } while (0)
#define TRRD(dst, addr, off) asm volatile("ds_read_b64_tr_b16 %0, %1 offset:%2" : "=v"(dst) : "v"(addr), "i"(off) : "memory")
#define LDB(dst, b, h) do { _Pragma("unroll") for (int n = 0; n < 2; ++n) _Pragma("unroll") for (int k = 0; k < 2; ++k) { \
        s16x4 lo, hi; TRRD(lo, boff[n], ((b) * 2 + (h)) * HTB + k * 8192); TRRD(hi, boff[n], ((b) * 2 + (h)) * HTB + k * 8192 + 1024); \
        dst[n][k] = __builtin_shufflevector(lo, hi, 0, 1, 2, 3, 4, 5, 6, 7); } } while (0)
#define MMA_K(ai, bj, At, Bt, k) do { _Pragma("unroll") for (int m = 0; m < 4; ++m) _Pragma("unroll") for (int n = 0; n < 2; ++n) \
        acc[ai][bj][m][n] = __builtin_amdgcn_mfma_f32_16x16x32_bf16(Bt[n][k], At[m][k], acc[ai][bj][m][n], 0, 0, 0); } while (0)
#define MMA(ai, bj, At, Bt, MID) do { __builtin_amdgcn_s_setprio(1); MMA_K(ai, bj, At, Bt, 0); SCHED; MID; SCHED; MMA_K(ai, bj, At, Bt, 1); __builtin_amdgcn_s_setprio(0); } while (0)
#define WAIT_V(n) asm volatile("s_waitcnt vmcnt(" #n ")" ::: "memory")
#define WAIT_VC(n2, n3) do { if (!comm) { if (helper) WAIT_V(n3); else WAIT_V(n2); } } while (0)
#define WAIT_L(n) asm volatile("s_waitcnt lgkmcnt(" #n ")" ::: "memory")
#define BAR __builtin_amdgcn_s_barrier()
#define SCHED __builtin_amdgcn_sched_barrier(0)

#define STAMP() do {} while (0)
#define STAMP2(i) do {} while (0)
    STAMP();
    f32x4 acc[2][2][4][2];
#pragma unroll
    for (int a = 0; a < 2; ++a)
#pragma unroll
        for (int b = 0; b < 2; ++b)
#pragma unroll
            for (int m = 0; m < 4; ++m)
#pragma unroll
                for (int n = 0; n < 2; ++n) acc[a][b][m][n] = (f32x4){0.f, 0.f, 0.f, 0.f};
    bf16x8 At[4][2], B0[2][2], B1[2][2];
    const char* cB = (const char*)P.w + (size_t)pn * 2 * hstepB;

#define AGG_PD 3
    const unsigned pfo = (unsigned)((((int)blockIdx.x >> 3) * 4 + (lane >> 4)) * N * 2 + (pn & ~3) * 512 + (lane & 15) * 128);
#define PFW(i) __builtin_amdgcn_global_load_lds((const unsigned*)((const char*)P.w + (size_t)(i) * 128 * N * 2 + pfo), (LAS unsigned*)(lds + PFBUF), 4, 0, 0)
    const int pi = (int)blockIdx.x >> 1, ph = (int)blockIdx.x & 1;
    const int npeer = ph ? 3 : 4;
    const unsigned xrow = (unsigned)(8 * (pi & 63) + (lane >> 3)), xcol = (unsigned)((pi >> 6) * 128 + (lane & 7) * 16);
    unsigned xo = xrow * RSR + xcol;
    const char* xsrc = (const char*)P.x + xrow * (unsigned)(K * 2) + xcol;
    unsigned* const ctr = (unsigned*)(myws + CTL_OFF);
    unsigned* const tmo = (unsigned*)(myws + CTL_OFF + 256);
#define XLOAD(c) __builtin_amdgcn_global_load_lds((const unsigned*)(xsrc + (size_t)(c) * 256), (LAS unsigned*)(lds + XBUF + ((c) & 3) * 1024), 16, 0, 0)

    unsigned cv;
    asm volatile("s_nop 4\n\tglobal_load_dword %0, %1, %2 sc1" : "=v"(cv) : "v"(0u), "s"(ctr) : "memory");
    if (comm) { XLOAD(0); XLOAD(1); XLOAD(2); WAIT_V(0); }
    else if (wid == 6) {
        WAIT_V(0);
    }
    else if (wid < AGG_PD) { PFW(wid + 1); WAIT_V(1); }
    else WAIT_V(0);
    asm volatile("" : "+v"(cv));
    STAMP();
    const unsigned call = (unsigned)__builtin_amdgcn_readfirstlane(cv) / NWG + 1u;
    const unsigned fbase = call * 64u;
    const size_t par_off = (size_t)(call & 1u) * GATHER_BYTES;
    const size_t slot_off = par_off + (size_t)rank * MSH * RSR;
    const int rot = ph ? pi % 3 : pi & 3;
    const char* pd[4];
#pragma unroll
    for (int k = 0; k < 4; ++k) { int q = k + rot; if (q >= npeer) q -= npeer; if (q >= npeer) q = 0;
        pd[k] = (const char*)P.ws[(rank + 1 + ph * 4 + q) & (NR - 1)] + slot_off; }
    const char* pflag;
    { const char* b = (const char*)P.ws[(rank + 1 + ph * 4) & (NR - 1)];
#pragma unroll
      for (int i = 1; i < 4; ++i) b = (lane == i && i < npeer) ? (const char*)P.ws[(rank + 1 + ph * 4 + i) & (NR - 1)] : b;
      pflag = b + FLAG_OFF + (size_t)rank * 512 + (size_t)pi * 4; }
    const unsigned long long fmask = ph ? 7ull : 15ull;
    const char* qflag = (const char*)myws + FLAG_OFF + (size_t)src * 512 + (size_t)(lane >> 5) * 256 + (size_t)(pm & 1) * 128 + (size_t)(lane & 31) * 4;
    const unsigned lds0 = (unsigned)(size_t)lds;
    u32x4 xv = (u32x4){0u, 0u, 0u, 0u};
    unsigned pf = 0, tnow = 0;

    unsigned e_iss = 0, e_i0 = 0, e_i1 = 0, e_due = 0;
#define AGG_TAU 275u
    int e_np = 0, e_r = 0, e_ack = 0, e_have = (remote && (comm || poller)) ? 0 : 1000, e_havo = e_have;
    int e_act = 0, timed_out = 0;
#define VMCNT_NOW(dst) do { unsigned _v; asm volatile("s_getreg_b32 %0, hwreg(HW_REG_IB_STS, 0, 24)" : "=s"(_v) :: "memory"); dst = (_v & 15u) | ((_v >> 18) & 48u); } while (0)
#define DECIDE(lim, now) do { unsigned _c; VMCNT_NOW(_c); const unsigned _done = e_iss - _c; \
        if (e_np > 0 && (int)(_done - e_i0) >= 0) { STAMP2(e_ack); ++e_ack; --e_np; e_i0 = e_i1; const unsigned _fv = fbase + (unsigned)e_ack; \
            asm volatile("s_mov_b64 exec, %2\n\tglobal_store_dword %0, %1, off sc0 sc1\n\ts_mov_b64 exec, -1" :: "v"(pflag), "v"(_fv), "s"(fmask) : "memory"); ++e_iss; } \
        e_act = (int)((((unsigned)(e_r - (lim))) & ((unsigned)(e_np - 2)) & ~((now) - e_due)) >> 31); \
    } while (0)
#define NOW() ((unsigned)__builtin_amdgcn_s_memrealtime())
#define PSLEEP do {} while (0)
#define AGG_OST 3
#define OSTORE(p, w) asm volatile("global_store_dwordx4 %0, %1, off sc0 sc1\n\ts_nop 1" :: "v"(p), "v"(w) : "memory")
#define POLL() __builtin_amdgcn_global_load_lds((const unsigned*)qflag, (LAS unsigned*)(lds + PBUF), 4, 0, 17)
#define PREAD() asm volatile("ds_read_b32 %0, %1" : "=v"(pf) : "v"(lds0 + PBUF + (unsigned)lane * 4u) : "memory")
#define PEVAL() do { asm volatile("" : "+v"(pf)); const unsigned _nE = fbase + (unsigned)e_have, _nO = fbase + (unsigned)e_havo; \
        const unsigned _b1 = (unsigned)__ballot(pf > _nE), _b2 = (unsigned)__ballot(pf > _nE + 1u); \
        const unsigned _b3 = (unsigned)(__ballot(pf > _nO) >> 32), _b4 = (unsigned)(__ballot(pf > _nO + 1u) >> 32); \
        INCIF(e_have, _b1); INCIF(e_have, _b2); \
        STAMP2O(_b3, _b4); INCIF(e_havo, _b3); INCIF(e_havo, _b4); } while (0)
#define INCIF(cnt, m) asm volatile("s_cmp_eq_u32 %1, -1\n\ts_addc_u32 %0, %0, 0" : "+s"(cnt) : "s"(m) : "scc")
#define STAMP2O(b3, b4) do {} while (0)
#define EVAL() do { PREAD(); WAIT_L(0); PEVAL(); } while (0)
#define XREAD(c) asm volatile("ds_read_b128 %0, %1" : "=v"(xv) : "v"(lds0 + XBUF + (unsigned)((c) & 3) * 1024u + (unsigned)lane * 16u) : "memory")
#define PST(k) do { asm volatile("s_nop 4\n\tglobal_store_dwordx4 %0, %1, %2 sc0 sc1\n\ts_nop 1" :: "v"(xo), "v"(xv), "s"(pd[k]) : "memory"); ++e_iss; } while (0)
#define SLOT(k) do { asm volatile("" : "+s"(e_act)); if (e_act != 0) { \
        if ((k) == 0 && e_r + 3 < NCH) { XLOAD(e_r + 3); ++e_iss; } \
        if ((k) < 3 || npeer == 4) PST(k); \
        if ((k) == 3) { if (e_np == 0) e_i0 = e_iss; e_i1 = e_iss; ++e_np; ++e_r; e_due += (e_r <= AGG_N1) ? AGG_TAU1 : AGG_TAU; xo += 256u; e_act = 0; XREAD(e_r); } } } while (0)
#define TICK(lim) do { DECIDE(lim, NOW()); WAIT_L(0); SLOT(0); SLOT(1); SLOT(2); SLOT(3); } while (0)
#define BOUNDED_UNTIL(cond, BODY) do { if (!(cond)) { const unsigned long long _t0 = __builtin_amdgcn_s_memrealtime(); unsigned _sp = 0; \
        while (!(cond)) { BODY; \
            if ((++_sp & 63u) == 0u) { if (__builtin_amdgcn_s_memrealtime() - _t0 > DEADLINE_TICKS) { timed_out = 1; break; } } \
            __builtin_amdgcn_s_sleep(1); } \
        if (timed_out != 0) { if (lane == 0) asm volatile("s_nop 4\n\tglobal_store_dword %0, %1, %2 sc1\n\tds_write_b32 %3, %1\n\ts_waitcnt vmcnt(0) lgkmcnt(0)" :: "v"(0u), "v"(call), "s"(tmo), "v"(lds0 + TFLAG) : "memory"); ++e_iss; e_have = 1000; e_havo = 1000; } } } while (0)
#define GATE(ev, need, lim) do { if (ev < (need)) { \
        if (GROLE == 3) { SLOT(2); SLOT(3); BOUNDED_UNTIL(ev >= (need), { EVAL(); TICK(lim); }); } \
        else { PGATE(ev, need); } } } while (0)
#define AGG_PPI 20
#define PGATE(ev, need) do { unsigned _tp = NOW() - 1000u; BOUNDED_UNTIL(ev >= (need), { if (NOW() - _tp > (unsigned)AGG_PPI) { POLL(); _tp = NOW(); } EVAL(); }); } while (0)

    if (poller) { asm volatile("ds_write_b32 %0, %1\n\ts_waitcnt lgkmcnt(0)" :: "v"(lds0 + PBUF + (unsigned)lane * 4u), "v"(0u) : "memory"); }
#define AGG_N1 0
#define AGG_TAU1 AGG_TAU
#define AGG_PHI 256
    e_due = (unsigned)__builtin_amdgcn_s_memrealtime() + (unsigned)blockIdx.x * AGG_TAU1 * AGG_PHI / 65536u;
    if (comm) { asm volatile("ds_write_b64 %0, %1" :: "v"(lds0 + TFLAG), "v"((u32x2){0u, 0u}) : "memory"); XREAD(0); TICK(DLA); WAIT_L(0); }
    BAR;
    STAGE_B(SB(0, 0), cB); HELP_B(SB(0, 0), cB); STAGE_B(SB(0, 1), cB + hstepB); HELP_B(SB(0, 1), cB + hstepB);
    STAGE_B(SB(1, 0), cB + kstepB); HELP_B(SB(1, 0), cB + kstepB); STAGE_B(SB(1, 1), cB + hstepB + kstepB); HELP_B(SB(1, 1), cB + hstepB + kstepB);
#define GROLE (comm ? 3 : 2)
    GATE(e_have, 1, DLA);
#define AGG_G6 1
#define AGG_PP 4
#define G2VAR e_have
#define LGATE(ev, need, lim) do { if (ev < (need)) { SLOT(2); SLOT(3); unsigned _tp = NOW() - 1000u; \
        BOUNDED_UNTIL(ev >= (need), { EVAL(); if (ev < (need)) { if (NOW() - _tp > 40u) { POLL(); ++e_iss; _tp = NOW(); } TICK(lim); } }); } } while (0)
#define AGG_LW 2
    if (comm && !remote) BOUNDED_UNTIL(e_ack >= AGG_LW, TICK(DLA));
    STAMP();
    const char* cA = (remote ? (const char*)myws + par_off + (size_t)src * MSH * RSR : (const char*)P.x) + (size_t)(pm & 1) * 2 * hstepA;
    BAR;
    STAGE_A(SA(0, 0), cA); HELP_A(SA(0, 0), cA); STAGE_A(SA(0, 1), cA + hstepA); HELP_A(SA(0, 1), cA + hstepA);
    GATE(e_havo, 1, DLA);
#undef GROLE
#undef GATE
    BAR;
    STAGE_A(SA(1, 0), cA + kstepA); HELP_A(SA(1, 0), cA + kstepA);
    if (poller && remote) { POLL(); WAIT_V(3); } else WAIT_VC(2, 3);
    if (!comm) { if (lane == 0) asm volatile("ds_add_u32 %0, %1" :: "v"(lds0 + TFLAG + 4), "v"(1u) : "memory"); }
    else { unsigned cnt = 0; BOUNDED_UNTIL(cnt >= 7u, { unsigned _cv; asm volatile("ds_read_b32 %0, %1\n\ts_waitcnt lgkmcnt(0)" : "=v"(_cv) : "v"(lds0 + TFLAG + 4) : "memory"); cnt = (unsigned)__builtin_amdgcn_readfirstlane(_cv); TICK(DLA); }); }
    BAR;
    if (comm) { tnow = NOW(); }
    if (wr == 1) BAR;
    STAMP();
#define RSTAGE_A(bufoff, gbase) do { if (ROLE != 3) { _Pragma("unroll") for (int _i = 0; _i < 2; ++_i) GLDS((const char*)(gbase) + voffA[_i], lds + (bufoff) + ldsw + _i * 8192); } } while (0)
#define RSTAGE_B(bufoff, gbase) do { if (ROLE != 3) { _Pragma("unroll") for (int _i = 0; _i < 2; ++_i) GLDS((const char*)(gbase) + (size_t)_i * 32 * N * 2 + voffB, lds + (bufoff) + ldsw + _i * 8192); } } while (0)
#define RHELP_A(bufoff, gbase) do { if (ROLE == 1) GLDS((const char*)(gbase) + hA + voffA[0], lds + (bufoff) + hL); } while (0)
#define RHELP_B(bufoff, gbase) do { if (ROLE == 1) GLDS((const char*)(gbase) + hB + voffB, lds + (bufoff) + hL); } while (0)
#define RWAIT() do { if (ROLE == 1) WAIT_V(8); else if (ROLE != 3) WAIT_V(6); } while (0)
#define RSLOT(k) do { if (ROLE == 3) SLOT(k); } while (0)
    auto kloop = [&](auto role_c, const int t_beg, const int t_end) __attribute__((always_inline)) {
      constexpr bool EAGER = decltype(role_c)::value == 4;
      constexpr int ROLE = EAGER ? 3 : decltype(role_c)::value;
      for (int t = t_beg; t < t_end; t += 2) {
        const int it = t >> 1;
        const bool last = (t == nt - 2);
        const char* a1 = cA + (size_t)(t >> 1) * CS + kstepA;
        const int t2 = last ? 0 : t + 2;
        const char* a2 = cA + (size_t)(t2 >> 1) * CS; const char* b2 = cB + (size_t)t2 * kstepB;
        const char* a3 = a2 + kstepA; const char* b3 = b2 + kstepB;
        const int want = (it + 4 < NCH) ? it + 4 : NCH;
        const int need = (it + 2 < NCH) ? it + 2 : NCH;
        const int lim = (it + DLA < NCH) ? it + DLA : NCH;
        const int lim8 = (it + 1 + DLA < NCH) ? it + 1 + DLA : NCH;
#define AGG_VA 0
#define AGG_VB 1
#define ASLOT(k) do { if (ROLE == 3 && !EAGER) SLOT(k); } while (0)
#define EPUSH_OK() ((int)((tnow - e_due) | ~(((unsigned)(e_r - lim)) & ((unsigned)(e_np - 2)))))
#define ETICK() do { if (EAGER) { WAIT_L(0); unsigned _c2; VMCNT_NOW(_c2); \
            const int _y = e_np > 0 ? (int)(e_iss - _c2 - e_i0) : -1; \
            if ((EPUSH_OK() & _y) >= 0) { \
                if (_y >= 0) { STAMP2(e_ack); ++e_ack; --e_np; e_i0 = e_i1; const unsigned _fv = fbase + (unsigned)e_ack; \
                    asm volatile("s_mov_b64 exec, %2\n\tglobal_store_dword %0, %1, off sc0 sc1\n\ts_mov_b64 exec, -1" :: "v"(pflag), "v"(_fv), "s"(fmask) : "memory"); ++e_iss; } \
                if (EPUSH_OK() >= 0) { \
                    XLOAD((e_r + 3 < NCH) ? e_r + 3 : NCH - 1); \
                    asm volatile("s_nop 4\n\tglobal_store_dwordx4 %0, %1, %2 sc0 sc1\n\tglobal_store_dwordx4 %0, %1, %3 sc0 sc1\n\tglobal_store_dwordx4 %0, %1, %4 sc0 sc1\n\ts_nop 1" \
                                 :: "v"(xo), "v"(xv), "s"(pd[0]), "s"(pd[1]), "s"(pd[2]) : "memory"); e_iss += 4; \
                    if (npeer == 4) PST(3); \
                    e_i0 = (e_np == 0) ? e_iss : e_i0; e_i1 = e_iss; ++e_np; ++e_r; e_due += (e_r <= AGG_N1) ? AGG_TAU1 : AGG_TAU; xo += 256u; XREAD(e_r); } } \
            tnow = NOW(); } } while (0)
        ETICK();
        if (ROLE == 3) WAIT_L(0);
        if (ROLE == 3 && !AGG_VB) PREAD();
        LDB(B0, 0, 0); SCHED; LDA(At, 0, 0); RSTAGE_A(SA(1, 1), a1 + hstepA);
        ASLOT(0);
        WAIT_L(8); BAR; WAIT_L(0); SCHED; MMA(0, 0, At, B0, RHELP_A(SA(1, 1), a1 + hstepA)); BAR; SCHED;
        ETICK();
        if (ROLE == 3 && AGG_VB) PREAD();
        LDB(B1, 0, 1); RSTAGE_B(SB(0, 0), b2);
        ASLOT(1);
        if (ROLE == 3) { if (AGG_VB) WAIT_L(8); if (e_havo < want) PEVAL();
          LGATE(G2VAR, need, lim);
        }
        BAR; WAIT_L(0); SCHED; MMA(0, 1, At, B1, RHELP_B(SB(0, 0), b2)); BAR;
        ETICK();
        LDA(At, 0, 1); RSTAGE_A(SA(0, 0), a2);
        ASLOT(2); if (ROLE == 3 && AGG_G6 && AGG_PP == 3) { if (e_havo < need) { POLL(); ++e_iss; } }
        if (ROLE == 3 && !EAGER) { tnow = NOW(); if (last) { __builtin_amdgcn_global_load_lds((const unsigned*)tmo, (LAS unsigned*)(lds + TBUF), 4, 0, 17); ++e_iss; } }
        BAR; WAIT_L(0); SCHED; MMA(1, 0, At, B0, RHELP_A(SA(0, 0), a2)); BAR; SCHED;
        ETICK();
        RSTAGE_B(SB(0, 1), b2 + hstepB);
        if (ROLE == 3 && AGG_G6 && AGG_PP == 4) { if (e_havo < need) { POLL(); ++e_iss; } }
        ASLOT(3); if (ROLE == 3 && !EAGER) DECIDE(lim, tnow);
        RWAIT();
        BAR; MMA(1, 1, At, B1, RHELP_B(SB(0, 1), b2 + hstepB)); BAR;
        ETICK();
        if (ROLE == 3) WAIT_L(0);
        LDB(B0, 1, 0); SCHED; LDA(At, 1, 0); RSTAGE_A(SA(0, 1), a2 + hstepA);
        ASLOT(0);
        WAIT_L(8); BAR; WAIT_L(0); SCHED; MMA(0, 0, At, B0, RHELP_A(SA(0, 1), a2 + hstepA)); BAR; SCHED;
        ETICK();
        if (ROLE == 3 && AGG_G6) PREAD();
        LDB(B1, 1, 1); RSTAGE_B(SB(1, 0), b3);
        ASLOT(1);
        if (ROLE == 3 && AGG_G6) { if (e_havo < need) { WAIT_L(8); PEVAL();
          LGATE(e_havo, need, lim);
        } }
        if (ROLE == 3) { if (it + 1 + AGG_PD < NCH) { PFW(it + 1 + AGG_PD); ++e_iss; } }
        BAR; WAIT_L(0); SCHED; MMA(0, 1, At, B1, RHELP_B(SB(1, 0), b3)); BAR;
        ETICK();
        LDA(At, 1, 1); RSTAGE_A(SA(1, 0), a3);
        ASLOT(2); if (ROLE == 3) { if (AGG_VB) { if (e_havo < want + 1) { POLL(); ++e_iss; } } if (!EAGER) tnow = NOW(); }
        BAR; WAIT_L(0); SCHED; MMA(1, 0, At, B0, RHELP_A(SA(1, 0), a3)); BAR; SCHED;
        ETICK();
        RSTAGE_B(SB(1, 1), b3 + hstepB);
        ASLOT(3); if (ROLE == 3 && !EAGER) { DECIDE(lim8, tnow); if (last) WAIT_V(0); }
        RWAIT();
        BAR; MMA(1, 1, At, B1, RHELP_B(SB(1, 1), b3 + hstepB)); BAR;
        STAMP();
      }
    };
#define AGG_NE 16
    if (comm) { kloop(IC<4>{}, 0, 2 * AGG_NE); kloop(IC<3>{}, 2 * AGG_NE, nt); } else if (helper) kloop(IC<1>{}, 0, nt); else kloop(IC<0>{}, 0, nt);
    if (wr == 0) BAR;
    if (comm) { WAIT_L(0); SLOT(0); SLOT(1); SLOT(2); SLOT(3); BOUNDED_UNTIL(e_ack >= NCH, TICK(NCH)); }
    if (tid == 0) __hip_atomic_fetch_add(ctr, 1u, __ATOMIC_RELAXED, __HIP_MEMORY_SCOPE_AGENT);
    {
        const unsigned tv0 = *(volatile LAS unsigned*)(lds + TFLAG), tv1 = *(volatile LAS unsigned*)(lds + TBUF);
        const bool bad = ((unsigned)__builtin_amdgcn_readfirstlane(tv0) == call) || ((unsigned)__builtin_amdgcn_readfirstlane(tv1) == call);
        const int row0 = pm * BM + wr * 64 + fr, col0 = pn * BM + wc * 32 + (fq & 1) * 16 + (fq >> 1) * 8;
        if (!bad) {
#pragma unroll
        for (int ai = 0; ai < 2; ++ai)
#pragma unroll
            for (int m = 0; m < 4; ++m) { bf16_t* rowp = P.out + (size_t)(row0 + ai * HALF + m * 16) * N + col0;
#pragma unroll
                for (int bj = 0; bj < 2; ++bj) { const f32x4 v0 = acc[ai][bj][m][0], v1 = acc[ai][bj][m][1];
                    const unsigned ax = silu2_bf16(v0[0], v0[1]), ay = silu2_bf16(v0[2], v0[3]);
                    const unsigned bx = silu2_bf16(v1[0], v1[1]), by = silu2_bf16(v1[2], v1[3]);
                    const u32x2 sx = __builtin_amdgcn_permlane16_swap(ax, bx, false, false), sy = __builtin_amdgcn_permlane16_swap(ay, by, false, false);
                    u32x4 w; w.x = sx.x; w.y = sy.x; w.z = sx.y; w.w = sy.y;
                    OSTORE(rowp + bj * HALF, w); } }
        } else {
            const u32x4 w = {0x7fc07fc0u, 0x7fc07fc0u, 0x7fc07fc0u, 0x7fc07fc0u};
            for (int i = 0; i < 16; ++i) *(u32x4*)(P.out + (size_t)(row0 + (i >> 3) * HALF + ((i >> 1) & 3) * 16) * N + col0 + (i & 1) * HALF) = w;
        }
    }
    WAIT_V(0);
}
}

extern "C" void kernel_launch(int rank, int nranks, ncclComm_t comm,
                              hipStream_t stream,
                              void* const* d_in, const int* in_sizes, int n_in,
                              void* d_out, int out_size,
                              const int* in_dtypes, int out_dtype,
                              void* const* d_out_all,
                              void* const* d_ws_all, size_t ws_bytes) {
    using namespace agg;
    (void)hipFuncSetAttribute((const void*)ag_gemm_k, hipFuncAttributeMaxDynamicSharedMemorySize, STAGE_BYTES);
    if (nranks != NR || ws_bytes < WS_NEED) { fprintf(stderr, "unsupported configuration\n"); return; }
    Params p;
    p.x = (const bf16_t*)d_in[0]; p.w = (const bf16_t*)d_in[1]; p.out = (bf16_t*)d_out; p.rank = rank;
    for (int i = 0; i < NR; ++i) p.ws[i] = (unsigned char*)d_ws_all[i];
    ag_gemm_k<<<NWG, 512, STAGE_BYTES, stream>>>(p);
}
```
